```python
import numpy as np

kernel_name = "gpu_gemm_rs_m4096_k4096_n16384_e5m2e5m2_dqrelu"


ABSMAX_RTOL = 0.05

COMPUTE_DTYPE = "float32"
EPILOGUE = "dequant_relu"
IN_QUANT = None
OUT_QUANT = None
IN_FP8 = "e5m2e5m2"
IN_DTYPES = {"x": "e5m2", "w_mat": "e5m2",
             "scale_x": "float32", "scale_w": "float32"}
WORLD_SIZE = 8

def _snap_fp8(x, mant_bits, max_abs, min_normal):
    a = np.abs(x.astype(np.float64))
    _m, _e = np.frexp(a)
    q = np.where(a >= min_normal, np.ldexp(1.0, _e - 1 - mant_bits),
                 min_normal * 2.0 ** -mant_bits)
    snapped = np.minimum(np.round(a / q) * q, max_abs)
    return (np.sign(x) * snapped).astype(np.float32)

def setup_inputs():

    x = _snap_fp8(np.random.randn(4096, 4096).astype(np.float32), 2, 57344.0, 2.0 ** -14)
    w_mat = _snap_fp8(np.random.randn(4096, 16384).astype(np.float32), 2, 57344.0, 2.0 ** -14)
    scale_x = (0.02 * (1.0 + np.random.rand(1))).astype(np.float32)
    scale_w = (0.02 * (1.0 + np.random.rand(1))).astype(np.float32)
    return {"x": x, "w_mat": w_mat, "scale_x": scale_x, "scale_w": scale_w}

def reference(x, w_mat, scale_x, scale_w):
    acc = (x @ w_mat).astype(np.float32)
    y = (acc * (scale_x * scale_w)).astype(np.float32)
    return np.maximum(y, 0.0).astype(np.float32)
```

```cpp
#include <hip/hip_runtime.h>
#include <rccl/rccl.h>
#include <cstdio>
#include <cstdint>

#define HIP_CHECK(expr) do { hipError_t _e = (expr); if (_e != hipSuccess) { \
    fprintf(stderr, "HIP error %s at %s:%d\n", hipGetErrorString(_e), __FILE__, __LINE__); \
    return; } } while (0)

typedef unsigned u32x4 __attribute__((ext_vector_type(4)));
typedef int i32x8 __attribute__((ext_vector_type(8)));
typedef float f32x16 __attribute__((ext_vector_type(16)));
typedef float f32x4 __attribute__((ext_vector_type(4)));
typedef __amdgpu_buffer_rsrc_t rsrc_t;

constexpr int NR = 8;
constexpr int GN = 16384;
constexpr int KS = 512;
constexpr int NC = 2048;
constexpr int NWG = 256;
constexpr int NPUSH = 32;
constexpr unsigned X_BYTES = 4096u * KS;
constexpr unsigned W_BYTES = (unsigned)KS * GN;
constexpr unsigned OUT_BYTES = 512u * GN * 4u;

constexpr unsigned OFF_LX = 0;
constexpr unsigned OFF_LW = 16u << 20;
constexpr unsigned OFF_FX = 24u << 20;
constexpr unsigned OFF_FW = OFF_FX + 8u * 128u * 128u;
constexpr unsigned OFF_FO = OFF_FW + 8u * 64u * 128u;
constexpr unsigned OFF_CNT = OFF_FO + 8u * 256u * 128u;
constexpr unsigned OFF_TMO = OFF_CNT + 256u * 128u;
constexpr unsigned OFF_DBG = OFF_TMO + 128u;
constexpr unsigned WS_USED = OFF_DBG + 256u * 8u * 16u * 8u;

constexpr unsigned long long DEADLINE_TICKS = 50000000ull;
constexpr int AUX_SYS = 17;
constexpr unsigned LDS_BYTES = 96u * 1024u;

struct Params {
    const unsigned char* x;
    const unsigned char* w;
    const float* sx;
    const float* sw;
    char* out[NR];
    char* ws[NR];
    int rank;
};

__device__ __forceinline__ rsrc_t mk_rsrc(const void* p, unsigned bytes) {
    unsigned long long a = (unsigned long long)p;
    unsigned lo = __builtin_amdgcn_readfirstlane((unsigned)a);
    unsigned hi = __builtin_amdgcn_readfirstlane((unsigned)(a >> 32));
    return __builtin_amdgcn_make_buffer_rsrc((void*)(((unsigned long long)hi << 32) | lo), 0, bytes, 0x00020000);
}

#define STAMP(idx) do { } while (0)

__device__ __forceinline__ bool wait_flags(char* ws, unsigned foff, bool active, unsigned call) {
    const unsigned* f = (const unsigned*)(ws + foff);
    unsigned* tmo = (unsigned*)(ws + OFF_TMO);
    unsigned v = active ? __hip_atomic_load(f, __ATOMIC_RELAXED, __HIP_MEMORY_SCOPE_SYSTEM) : call;
    if (__all(v >= call)) return true;
    const unsigned long long t0 = __builtin_amdgcn_s_memrealtime();
    for (;;) {
        __builtin_amdgcn_s_sleep(4);
        v = active ? __hip_atomic_load(f, __ATOMIC_RELAXED, __HIP_MEMORY_SCOPE_SYSTEM) : call;
        if (__all(v >= call)) return true;
        const unsigned t = __hip_atomic_load(tmo, __ATOMIC_RELAXED, __HIP_MEMORY_SCOPE_AGENT);
        if (t == call || __builtin_amdgcn_s_memrealtime() - t0 > DEADLINE_TICKS) {
            __hip_atomic_store(tmo, call, __ATOMIC_RELAXED, __HIP_MEMORY_SCOPE_AGENT);
            return false;
        }
    }
}

__device__ __forceinline__ void set_flag(char* ws_dst, unsigned foff, unsigned call, int lane) {
    asm volatile("s_waitcnt vmcnt(0)" ::: "memory");
    if (lane == 0) __hip_atomic_store((unsigned*)(ws_dst + foff), call, __ATOMIC_RELAXED, __HIP_MEMORY_SCOPE_SYSTEM);
}

__global__ void __launch_bounds__(512, 2) fused_gemm_rs(Params p) {
    extern __shared__ __attribute__((aligned(16))) char lds[];
    const int tid = threadIdx.x;
    const int lane = tid & 63;
    const int wave = __builtin_amdgcn_readfirstlane(tid >> 6);
    const int rank = p.rank;
    const int b = blockIdx.x;
    char* const ws = p.ws[rank];
    const int c32 = lane & 31, h = lane >> 5;

    unsigned* cnt = (unsigned*)(ws + OFF_CNT + (unsigned)b * 128u);
    const unsigned call = __builtin_amdgcn_readfirstlane(
        __hip_atomic_load(cnt, __ATOMIC_RELAXED, __HIP_MEMORY_SCOPE_AGENT)) + 1u;
    STAMP(0);

    if (b < NPUSH) {
        char* wsd = p.ws[wave];
        const rsrc_t rd = mk_rsrc(wsd, WS_USED);
        const rsrc_t rx = mk_rsrc(p.x, X_BYTES);
        const rsrc_t rw = mk_rsrc(p.w, W_BYTES);
        for (int n = b; n < 192; n += NPUSH) {
            if (n < 64) {
                const int pn = n >> 2, q = n & 3;
                const unsigned col = (unsigned)(wave * NC + pn * 128 + 4 * c32);
#pragma unroll
                for (int uu = 0; uu < 4; ++uu) {
                    const int u = q * 4 + uu;
                    const unsigned src = (unsigned)(32 * u + 16 * h) * GN + col;
                    unsigned d[16];
#pragma unroll
                    for (int i = 0; i < 16; ++i) d[i] = __builtin_amdgcn_raw_buffer_load_b32(rw, src + (unsigned)i * GN, 0, 0);
                    u32x4 o[4];
#pragma unroll
                    for (int g = 0; g < 4; ++g) {
                        const unsigned lo01 = __builtin_amdgcn_perm(d[4 * g + 1], d[4 * g], 0x05010400u);
                        const unsigned hi01 = __builtin_amdgcn_perm(d[4 * g + 1], d[4 * g], 0x07030602u);
                        const unsigned lo23 = __builtin_amdgcn_perm(d[4 * g + 3], d[4 * g + 2], 0x05010400u);
                        const unsigned hi23 = __builtin_amdgcn_perm(d[4 * g + 3], d[4 * g + 2], 0x07030602u);
                        o[0][g] = __builtin_amdgcn_perm(lo23, lo01, 0x05040100u);
                        o[1][g] = __builtin_amdgcn_perm(lo23, lo01, 0x07060302u);
                        o[2][g] = __builtin_amdgcn_perm(hi23, hi01, 0x05040100u);
                        o[3][g] = __builtin_amdgcn_perm(hi23, hi01, 0x07060302u);
                    }
                    const unsigned dst = OFF_LW + ((unsigned)rank << 20) + ((unsigned)pn << 16) + ((unsigned)u << 12) + (unsigned)lane * 16u;
#pragma unroll
                    for (int j = 0; j < 4; ++j) __builtin_amdgcn_raw_buffer_store_b128(o[j], rd, dst + (unsigned)j * 1024u, 0, AUX_SYS);
                }
                set_flag(wsd, OFF_FW + (unsigned)(rank * 64 + n) * 128u, call, lane);
            } else {
                const int m = n - 64;
                const int i = (m >> 6) * 8 + ((((m >> 3) & 7) + rank) & 7);
                const int rg = (m & 7) * 16 + i;
                const unsigned src = (unsigned)(32 * rg + c32) * KS + 16u * h;
                u32x4 v[16];
#pragma unroll
                for (int u = 0; u < 16; ++u) v[u] = __builtin_amdgcn_raw_buffer_load_b128(rx, src + 32u * u, 0, 0);
                const unsigned dst = OFF_LX + ((unsigned)rank << 21) + ((unsigned)rg << 14) + (unsigned)lane * 16u;
#pragma unroll
                for (int u = 0; u < 16; ++u) __builtin_amdgcn_raw_buffer_store_b128(v[u], rd, dst + (unsigned)u * 1024u, 0, AUX_SYS);
                set_flag(wsd, OFF_FX + (unsigned)(rank * 128 + rg) * 128u, call, lane);
            }
        }
    }
    STAMP(1);

    const int xg = b & 7, idx = 31 - (b >> 3);
    const int pn = (xg & 3) * 4 + (idx & 3);
    const int ib = (idx >> 2) * 2 + (xg >> 2);
    const int tile = ib * 16 + pn;
    const int rg = wave * 16 + ib;

    const rsrc_t rl = mk_rsrc(ws, WS_USED);
    f32x16 acc[4];
#pragma unroll
    for (int j = 0; j < 4; ++j)
#pragma unroll
        for (int e = 0; e < 16; ++e) acc[j][e] = 0.f;

    bool ok = true;
    auto src_of = [&](int n) -> int {
        if (n == 0) return rank;
        const int tr = (ib - rank) & 7;
        const int tau = (n - 1) < tr ? (n - 1) : n;
        return (ib - tau) & 7;
    };
    const unsigned a_base = OFF_LX + ((unsigned)rg << 14) + (unsigned)lane * 16u;
    const unsigned b_base = OFF_LW + ((unsigned)pn << 16) + (unsigned)wave * 2048u + (unsigned)lane * 16u;
    const unsigned fl_off = lane < 4 ? OFF_FW + (unsigned)(pn * 4 + lane) * 128u : OFF_FX + (unsigned)rg * 128u;
    const unsigned fl_str = lane < 4 ? 64u * 128u : 128u * 128u;
    char* const lw = lds + wave * 2048 + lane * 16;
    const char* const lr = lds + lane * 16;

    u32x4 A[4][4];
    u32x4 B[2][2];
#define LOAD_A(slot, s, q) do { const unsigned _o = a_base + ((unsigned)(s) << 21) + (unsigned)(q) * 4096u; \
        _Pragma("unroll") for (int _u = 0; _u < 4; ++_u) A[slot][_u] = __builtin_amdgcn_raw_buffer_load_b128(rl, _o + _u * 1024u, 0, AUX_SYS); } while (0)
#define LOAD_B(slot, s, q) do { const unsigned _o = b_base + ((unsigned)(s) << 20) + (unsigned)(q) * 16384u; \
        _Pragma("unroll") for (int _u = 0; _u < 2; ++_u) B[slot][_u] = __builtin_amdgcn_raw_buffer_load_b128(rl, _o + _u * 1024u, 0, AUX_SYS); } while (0)
#define WRITE_B(slot, buf) do { _Pragma("unroll") for (int _u = 0; _u < 2; ++_u) *(u32x4*)(lw + (buf) * 16384 + _u * 1024) = B[slot][_u]; } while (0)
#define COMPUTE(slot, buf) do { \
        _Pragma("unroll") for (int _t = 0; _t < 2; ++_t) { \
            i32x8 af; \
            _Pragma("unroll") for (int _e = 0; _e < 4; ++_e) { af[_e] = (int)A[slot][2 * _t][_e]; af[4 + _e] = (int)A[slot][2 * _t + 1][_e]; } \
            _Pragma("unroll") for (int _j = 0; _j < 4; ++_j) { \
                const u32x4 b0 = *(const u32x4*)(lr + (buf) * 16384 + (2 * _t) * 4096 + _j * 1024); \
                const u32x4 b1 = *(const u32x4*)(lr + (buf) * 16384 + (2 * _t + 1) * 4096 + _j * 1024); \
                i32x8 bf; \
                _Pragma("unroll") for (int _e = 0; _e < 4; ++_e) { bf[_e] = (int)b0[_e]; bf[4 + _e] = (int)b1[_e]; } \
                acc[_j] = __builtin_amdgcn_mfma_scale_f32_32x32x64_f8f6f4(af, bf, acc[_j], 1, 1, 0, 0, 0, 0); \
            } \
        } } while (0)

    int s_cur = src_of(0);
    ok &= wait_flags(ws, fl_off + (unsigned)s_cur * fl_str, lane < 5, call);
    STAMP(2);
    LOAD_B(0, s_cur, 0); LOAD_A(0, s_cur, 0);
    LOAD_B(1, s_cur, 1); LOAD_A(1, s_cur, 1);
    WRITE_B(0, 0);
    LOAD_B(0, s_cur, 2); LOAD_A(2, s_cur, 2);
    for (int n = 0; n < 8; ++n) {
        const bool more = n < 7;
        const int s_nxt = src_of(more ? n + 1 : n);
        __syncthreads();
        WRITE_B(1, 1);
        LOAD_B(1, s_cur, 3); LOAD_A(3, s_cur, 3);
        COMPUTE(0, 0);
        __syncthreads();
        WRITE_B(0, 0);
        if (more) {
            ok &= wait_flags(ws, fl_off + (unsigned)s_nxt * fl_str, lane < 5, call);
            STAMP(3 + n);
            LOAD_B(0, s_nxt, 0); LOAD_A(0, s_nxt, 0);
        }
        COMPUTE(1, 1);
        __syncthreads();
        WRITE_B(1, 1);
        if (more) { LOAD_B(1, s_nxt, 1); LOAD_A(1, s_nxt, 1); }
        COMPUTE(2, 0);
        __syncthreads();
        if (more) { WRITE_B(0, 0); LOAD_B(0, s_nxt, 2); LOAD_A(2, s_nxt, 2); }
        COMPUTE(3, 1);
        s_cur = s_nxt;
    }
    STAMP(10);

    {
        const unsigned t = __hip_atomic_load((unsigned*)(ws + OFF_TMO), __ATOMIC_RELAXED, __HIP_MEMORY_SCOPE_AGENT);
        if (t == call) ok = false;
        float scale = p.sx[0] * p.sw[0];
        if (!ok) scale = __builtin_nanf("");
        const rsrc_t ro = mk_rsrc(p.out[wave], OUT_BYTES);
        const unsigned o_base = ((unsigned)(ib * 32 + 4 * h) * GN + (unsigned)(rank * NC + pn * 128 + 4 * c32)) * 4u;
#pragma unroll
        for (int e = 0; e < 16; ++e) {
            u32x4 v;
#pragma unroll
            for (int j = 0; j < 4; ++j) {
                const float y = acc[j][e] * scale;
                v[j] = __builtin_bit_cast(unsigned, (y <= 0.f) ? 0.f : y);
            }
            const unsigned row = (unsigned)((e & 3) + 8 * (e >> 2));
            __builtin_amdgcn_raw_buffer_store_b128(v, ro, o_base + row * (GN * 4u), 0, AUX_SYS);
        }
        if (wave != rank) {
            set_flag(p.ws[wave], OFF_FO + (unsigned)(rank * 256 + tile) * 128u, call, lane);
            STAMP(11);
            const bool got = wait_flags(ws, OFF_FO + (unsigned)(wave * 256 + tile) * 128u, lane == 0, call);
            if (!got && lane == 0) {
                ((float*)p.out[rank])[(size_t)(ib * 32) * GN + wave * NC + pn * 128] = __builtin_nanf("");
            }
        }
        STAMP(12);
    }
    if (tid == 0) __hip_atomic_store(cnt, call, __ATOMIC_RELAXED, __HIP_MEMORY_SCOPE_AGENT);
}

extern "C" void kernel_launch(int rank, int nranks, ncclComm_t comm,
                              hipStream_t stream,
                              void* const* d_in, const int* in_sizes, int n_in,
                              void* d_out, int out_size,
                              const int* in_dtypes, int out_dtype,
                              void* const* d_out_all,
                              void* const* d_ws_all, size_t ws_bytes) {
    (void)comm; (void)in_sizes; (void)n_in; (void)d_out; (void)out_size; (void)in_dtypes; (void)out_dtype;
    if (nranks != NR || (unsigned)rank >= (unsigned)NR || ws_bytes < (size_t)WS_USED) {
        fprintf(stderr, "unsupported configuration\n");
        return;
    }
    Params p;
    p.x = (const unsigned char*)d_in[0];
    p.w = (const unsigned char*)d_in[1];
    p.sx = (const float*)d_in[2];
    p.sw = (const float*)d_in[3];
    for (int i = 0; i < NR; ++i) { p.out[i] = (char*)d_out_all[i]; p.ws[i] = (char*)d_ws_all[i]; }
    p.rank = rank;
    HIP_CHECK(hipFuncSetAttribute((const void*)fused_gemm_rs, hipFuncAttributeMaxDynamicSharedMemorySize, (int)LDS_BYTES));
    hipLaunchKernelGGL(fused_gemm_rs, dim3(NWG), dim3(512), LDS_BYTES, stream, p);
    HIP_CHECK(hipGetLastError());
}
```

```cpp
#include <hip/hip_runtime.h>
#include <rccl/rccl.h>
#include <cstdio>
#include <cstdint>

#define CWAVES 6
#define PSTEP 1
#define USPLIT 1
#define AUX_O 17
#define POLL_SLEEP 8

#define HIP_CHECK(expr) do { hipError_t _e = (expr); if (_e != hipSuccess) { \
    fprintf(stderr, "HIP error %s at %s:%d\n", hipGetErrorString(_e), __FILE__, __LINE__); \
    return; } } while (0)

typedef unsigned u32x4 __attribute__((ext_vector_type(4)));
typedef unsigned u32x2 __attribute__((ext_vector_type(2)));
typedef int i32x8 __attribute__((ext_vector_type(8)));
typedef float f32x16 __attribute__((ext_vector_type(16)));
typedef unsigned long long u64;
typedef __amdgpu_buffer_rsrc_t rsrc_t;

constexpr int NR = 8;
constexpr int GN = 16384;
constexpr int KS = 512;
constexpr int NC = 2048;
constexpr int NWG = 256;
constexpr int NH = 56;
constexpr int NG = NWG - NH;
constexpr int NCW = 8 * CWAVES;
constexpr int NITEM = 768;
constexpr int NJOB = NITEM + 256;
constexpr unsigned X_BYTES = 4096u * KS;
constexpr unsigned W_BYTES = (unsigned)KS * GN;
constexpr unsigned OUT_BYTES = 512u * GN * 4u;
constexpr unsigned SLOT = 512u + 16384u;

constexpr unsigned OFF_LX = 0;
constexpr unsigned OFF_LW = 16u << 20;
constexpr unsigned OFF_FX = 24u << 20;
constexpr unsigned OFF_FW = OFF_FX + 8u * 128u * 128u;
constexpr unsigned OFF_JF = OFF_FW + 8u * 16u * 128u;
constexpr unsigned OFF_UF = OFF_JF + 8u * 1024u * 64u;
constexpr unsigned OFF_CNT = OFF_UF + 8u * 256u * 128u;
constexpr unsigned OFF_TMO = OFF_CNT + 256u * 128u;
constexpr unsigned OFF_JTK = OFF_TMO + 128u;
constexpr unsigned OFF_QP = OFF_JTK + 8u * 128u;
constexpr unsigned OFF_DBG = OFF_QP + 8u * 128u;
constexpr unsigned OFF_SW = 26u << 20;
constexpr unsigned OFF_PK = 34u << 20;
constexpr unsigned OFF_RK = 68u << 20;
constexpr unsigned WS_USED = OFF_RK + 8u * 256u * SLOT;
static_assert(OFF_DBG + 256u * 8u * 16u * 8u <= OFF_SW, "layout");
static_assert(OFF_PK + 8u * 256u * SLOT <= OFF_RK, "layout");

constexpr u64 DEADLINE_TICKS = 50000000ull;
constexpr int AUX_SYS = 17;
constexpr unsigned LDS_BYTES = 8u * SLOT;

struct Params {
    const unsigned char* x;
    const unsigned char* w;
    const float* sx;
    const float* sw;
    char* out;
    char* ws[NR];
    int rank;
};

__device__ __forceinline__ rsrc_t mk_rsrc(const void* p, unsigned bytes) {
    u64 a = (u64)p;
    unsigned lo = __builtin_amdgcn_readfirstlane((unsigned)a);
    unsigned hi = __builtin_amdgcn_readfirstlane((unsigned)(a >> 32));
    return __builtin_amdgcn_make_buffer_rsrc((void*)(((u64)hi << 32) | lo), 0, __builtin_amdgcn_readfirstlane(bytes), 0x00020000);
}

#define STAMP(idx) do { } while (0)

__device__ __forceinline__ bool wait_ge(char* ws, unsigned foff, unsigned call) {
    const unsigned* f = (const unsigned*)(ws + foff);
    unsigned* tmo = (unsigned*)(ws + OFF_TMO);
    unsigned v = __hip_atomic_load(f, __ATOMIC_RELAXED, __HIP_MEMORY_SCOPE_SYSTEM);
    if (__all(v >= call)) return true;
    const u64 t0 = __builtin_amdgcn_s_memrealtime();
    for (unsigned it = 0;; ++it) {
        __builtin_amdgcn_s_sleep(POLL_SLEEP);
        if (v < call) v = __hip_atomic_load(f, __ATOMIC_RELAXED, __HIP_MEMORY_SCOPE_SYSTEM);
        if (__all(v >= call)) return true;
        if ((it & 63u) != 63u) continue;
        const unsigned t = __hip_atomic_load(tmo, __ATOMIC_RELAXED, __HIP_MEMORY_SCOPE_AGENT);
        if (t == call || __builtin_amdgcn_s_memrealtime() - t0 > DEADLINE_TICKS) {
            __hip_atomic_store(tmo, call, __ATOMIC_RELAXED, __HIP_MEMORY_SCOPE_AGENT);
            return false;
        }
    }
}
__device__ __forceinline__ unsigned wait_flag64(char* ws, unsigned foff, unsigned call) {
    const u64* f = (const u64*)(ws + foff);
    unsigned* tmo = (unsigned*)(ws + OFF_TMO);
    const u64 t0 = __builtin_amdgcn_s_memrealtime();
    for (unsigned it = 0;; ++it) {
        const u64 v = __hip_atomic_load(f, __ATOMIC_RELAXED, __HIP_MEMORY_SCOPE_SYSTEM);
        const unsigned hi = __builtin_amdgcn_readfirstlane((unsigned)(v >> 32));
        const unsigned lo = __builtin_amdgcn_readfirstlane((unsigned)v);
        if (hi >= call) return lo;
        __builtin_amdgcn_s_sleep(POLL_SLEEP);
        if ((it & 63u) != 63u) continue;
        const unsigned t = __hip_atomic_load(tmo, __ATOMIC_RELAXED, __HIP_MEMORY_SCOPE_AGENT);
        if (t == call || __builtin_amdgcn_s_memrealtime() - t0 > DEADLINE_TICKS) {
            __hip_atomic_store(tmo, call, __ATOMIC_RELAXED, __HIP_MEMORY_SCOPE_AGENT);
            return ~0u;
        }
    }
}

__device__ __forceinline__ void drain() {
    asm volatile("" ::: "memory");
    __builtin_amdgcn_s_waitcnt(0x0F70);
    asm volatile("s_waitcnt vmcnt(0)" ::: "memory");
}
__device__ __forceinline__ void store_flag(char* addr, unsigned call, unsigned payload) {
    __hip_atomic_store((u64*)addr, ((u64)call << 32) | payload, __ATOMIC_RELAXED, __HIP_MEMORY_SCOPE_SYSTEM);
}
__device__ __forceinline__ unsigned ticket(char* ws, unsigned off, int lane) {
    unsigned r = 0;
    if (lane == 0) r = __hip_atomic_fetch_add((unsigned*)(ws + off), 1u, __ATOMIC_RELAXED, __HIP_MEMORY_SCOPE_AGENT);
    return __builtin_amdgcn_readfirstlane(r);
}

__device__ __forceinline__ bool item_is_w(int n) { return n < 128 || (n >= 384 && n < 512); }
__device__ __forceinline__ int item_w_pn(int n, int src) {
    const int hi = n >= 384, pi = (((n & 127) >> 4) + src) & 7;
    return (pi >> 1) * 4 + hi * 2 + (pi & 1);
}
__device__ __forceinline__ int item_x_rg(int n, int src) {
    const int g = n >= 512, m = g ? n - 512 : n - 128;
    return (m & 7) * 16 + g * 8 + (((m >> 5) + src) & 7);
}

__global__ void __launch_bounds__(512, 2) fused_gemm_rs(Params p) {
    extern __shared__ __attribute__((aligned(16))) char lds[];
    const int tid = threadIdx.x;
    const int lane = tid & 63;
    const int wave = __builtin_amdgcn_readfirstlane(tid >> 6);
    const int rank = p.rank;
    const int b = blockIdx.x;
    char* const ws = p.ws[rank];
    const int c32 = lane & 31, h = lane >> 5;
    char* const pk = lds + wave * SLOT;

    unsigned* cnt = (unsigned*)(ws + OFF_CNT + (unsigned)b * 128u + (unsigned)wave * 4u);
    const unsigned call = __builtin_amdgcn_readfirstlane(
        __hip_atomic_load(cnt, __ATOMIC_RELAXED, __HIP_MEMORY_SCOPE_AGENT)) + 1u;
    if (lane == 0) __hip_atomic_store(cnt, call, __ATOMIC_RELAXED, __HIP_MEMORY_SCOPE_AGENT);
    STAMP(0);

    if (b < NH) {
        if (wave >= CWAVES) return;
        const int peer = (rank + 1 + b % 7) & 7;
        char* const wsp = p.ws[peer];
        const rsrc_t rf = mk_rsrc(ws + OFF_JF + (unsigned)peer * (1024u * 64u), 1024u * 64u);
        const unsigned tk = OFF_JTK + (unsigned)peer * 128u;
        const unsigned tbase = (call - 1u) * (unsigned)NJOB - (unsigned)NCW;
        unsigned t = (unsigned)((b / 7) * CWAVES + wave);
        u32x2 fv = u32x2{0u, 0u};
        unsigned pend = 0, pendv = 0;
        int si = 1;
        while (t < (unsigned)NJOB) {
            unsigned ent = __builtin_amdgcn_readfirstlane(fv[0]);
            if (__builtin_amdgcn_readfirstlane(fv[1]) < call) ent = wait_flag64(ws, OFF_JF + ((unsigned)peer * 1024u + t) * 64u, call);
            unsigned so, dof, nb, fo;
            if (t < (unsigned)NITEM) {
                nb = 4096u;
                if (item_is_w((int)t)) {
                    const unsigned o = ((unsigned)item_w_pn((int)t, peer) << 16) + ((t & 15u) << 12);
                    so = OFF_SW + ((unsigned)rank << 20) + o;
                    dof = OFF_LW + ((unsigned)peer << 20) + o;
                    fo = OFF_FW + (unsigned)peer * 2048u + (o >> 9);
                } else {
                    const unsigned rg = (unsigned)item_x_rg((int)t, peer), q = (t >> 3) & 3u;
                    so = dof = OFF_LX + ((unsigned)peer << 21) + (rg << 14) + (q << 12);
                    fo = OFF_FX + ((unsigned)peer * 128u + rg) * 128u + q * 8u;
                }
            } else {
                const unsigned k = t - (unsigned)NITEM;
                const unsigned cv = ent & 0xffffu;
                nb = cv <= 4096u ? (512u + cv * 4u + 63u) & ~63u : 0u;
                so = OFF_PK + ((unsigned)rank * 256u + ((ent >> 16) & 255u)) * SLOT;
                dof = OFF_RK + ((unsigned)peer * 256u + k) * SLOT;
                fo = OFF_UF + ((unsigned)peer * 256u + k) * 128u;
            }
            const rsrc_t rs = mk_rsrc(wsp + so, nb);
            const rsrc_t rd = mk_rsrc(ws + dof, nb);
            u32x4 d[PSTEP];
#pragma unroll
            for (int r = 0; r < PSTEP; ++r) d[r] = __builtin_amdgcn_raw_buffer_load_b128(rs, (unsigned)r * 1024u + (unsigned)lane * 16u, 0, AUX_SYS);
            const unsigned tn = ticket(ws, tk, lane) - tbase;
            drain();
            if (pend && lane == 0) store_flag(ws + pend, call, pendv);
            fv = __builtin_amdgcn_raw_buffer_load_b64(rf, tn * 64u, 0, AUX_SYS);
#pragma unroll
            for (int r = 0; r < PSTEP; ++r) __builtin_amdgcn_raw_buffer_store_b128(d[r], rd, (unsigned)r * 1024u + (unsigned)lane * 16u, 0, AUX_SYS);
            for (unsigned off = PSTEP * 1024u; off < nb; off += PSTEP * 1024u) {
#pragma unroll
                for (int r = 0; r < PSTEP; ++r) d[r] = __builtin_amdgcn_raw_buffer_load_b128(rs, off + (unsigned)r * 1024u + (unsigned)lane * 16u, 0, AUX_SYS);
#pragma unroll
                for (int r = 0; r < PSTEP; ++r) __builtin_amdgcn_raw_buffer_store_b128(d[r], rd, off + (unsigned)r * 1024u + (unsigned)lane * 16u, 0, AUX_SYS);
            }
            pend = fo; pendv = ent;
            t = tn;
        }
        drain();
        if (pend && lane == 0) store_flag(ws + pend, call, pendv);
        STAMP(15);
        return;
    }

    const int g = b - NH;
    {
        const rsrc_t rd = mk_rsrc(ws, OFF_PK);
        for (int j = g * 8 + wave; j < 2560; j += NG * 8) {
            const int half = j >= 1280, jj = j - half * 1280;
            u32x4 d[4];
            if (jj < 1024) {
                const rsrc_t rw = mk_rsrc(p.w, W_BYTES);
                const int n = half * 384 + (jj >> 3), dest = jj & 7;
                const int pn = item_w_pn(n, rank), u = n & 15;
                const unsigned src = (unsigned)(32 * u + 16 * h) * GN + (unsigned)(dest * NC + pn * 128 + 4 * c32);
                unsigned v[16];
#pragma unroll
                for (int i = 0; i < 16; ++i) v[i] = __builtin_amdgcn_raw_buffer_load_b32(rw, src + (unsigned)i * GN, 0, 0);
#pragma unroll
                for (int q = 0; q < 4; ++q) {
                    const unsigned lo01 = __builtin_amdgcn_perm(v[4 * q + 1], v[4 * q], 0x05010400u);
                    const unsigned hi01 = __builtin_amdgcn_perm(v[4 * q + 1], v[4 * q], 0x07030602u);
                    const unsigned lo23 = __builtin_amdgcn_perm(v[4 * q + 3], v[4 * q + 2], 0x05010400u);
                    const unsigned hi23 = __builtin_amdgcn_perm(v[4 * q + 3], v[4 * q + 2], 0x07030602u);
                    d[0][q] = __builtin_amdgcn_perm(lo23, lo01, 0x05040100u);
                    d[1][q] = __builtin_amdgcn_perm(lo23, lo01, 0x07060302u);
                    d[2][q] = __builtin_amdgcn_perm(hi23, hi01, 0x05040100u);
                    d[3][q] = __builtin_amdgcn_perm(hi23, hi01, 0x07060302u);
                }
                const unsigned o = ((unsigned)pn << 16) + ((unsigned)u << 12);
                const unsigned dst = (dest == rank ? OFF_LW : OFF_SW) + ((unsigned)dest << 20) + o + (unsigned)lane * 16u;
#pragma unroll
                for (int r = 0; r < 4; ++r) __builtin_amdgcn_raw_buffer_store_b128(d[r], rd, dst + (unsigned)r * 1024u, 0, AUX_SYS);
                char* f = dest == rank ? ws + OFF_FW + (unsigned)rank * 2048u + (o >> 9)
                                       : p.ws[dest] + OFF_JF + (unsigned)(rank * 1024 + n) * 64u;
                drain();
                if (lane == 0) store_flag(f, call, 0u);
            } else {
                const rsrc_t rx = mk_rsrc(p.x, X_BYTES);
                const int n = half * 384 + 128 + (jj - 1024);
                const int rg = item_x_rg(n, rank), q = (n >> 3) & 3;
                const unsigned src = (unsigned)(32 * rg + c32) * KS + 16u * h + 128u * q;
#pragma unroll
                for (int r = 0; r < 4; ++r) d[r] = __builtin_amdgcn_raw_buffer_load_b128(rx, src + 32u * r, 0, 0);
                const unsigned dst = OFF_LX + ((unsigned)rank << 21) + ((unsigned)rg << 14) + ((unsigned)q << 12) + (unsigned)lane * 16u;
#pragma unroll
                for (int r = 0; r < 4; ++r) __builtin_amdgcn_raw_buffer_store_b128(d[r], rd, dst + (unsigned)r * 1024u, 0, AUX_SYS);
                char* f = (lane & 7) == rank ? ws + OFF_FX + (unsigned)(rank * 128 + rg) * 128u + (unsigned)q * 8u
                                             : p.ws[lane & 7] + OFF_JF + (unsigned)(rank * 1024 + n) * 64u;
                drain();
                if (lane < 8) store_flag(f, call, 0u);
            }
        }
    }
    STAMP(1);
    const rsrc_t rl = mk_rsrc(ws, OFF_SW);
    const float scale_ok = p.sx[0] * p.sw[0];
    const rsrc_t ro = mk_rsrc(p.out, OUT_BYTES);
    const int xg = b & 7, li = g >> 3;
    char* const lw = lds + wave * 2048 + lane * 16;
    const char* const lr = lds + lane * 16;

    for (int round = 0; round < 2; ++round) {
        int ti = li;
        if (round) { if (li >= 7) break; ti = 25 + li; }
        const int kk = ti < 16 ? (ti >> 3) * 2 + (ti & 1) : (ti & 3);
        const int mm = ti < 16 ? (ti >> 1) & 3 : ti >> 2;
        const int pn = (xg & 3) * 4 + kk;
        const int ib = mm * 2 + (xg >> 2);
        const int tile = ib * 16 + pn;
        const int rg = wave * 16 + ib;
        const int key = (ti >> 3) == 1 ? (pn >> 2) * 2 + (pn & 1) : ib;

        f32x16 acc[4];
#pragma unroll
        for (int j = 0; j < 4; ++j)
#pragma unroll
            for (int e = 0; e < 16; ++e) acc[j][e] = 0.f;

        bool ok = true;
        const int tr = (key - rank) & 7;
        const unsigned a_base = OFF_LX + ((unsigned)rg << 14) + (unsigned)lane * 16u;
        const unsigned b_base = OFF_LW + ((unsigned)pn << 16) + (unsigned)wave * 2048u + (unsigned)lane * 16u;
        const unsigned fl_off = (lane & 4) == 0 ? OFF_FX + (unsigned)rg * 128u + (unsigned)(lane & 3) * 8u + 4u
                                                : OFF_FW + (unsigned)pn * 128u + (unsigned)(((lane & 3) * 4 + (wave >> 1)) * 8) + 4u;
        const unsigned fl_str = (lane & 4) == 0 ? 128u * 128u : 16u * 128u;

        u32x4 A[4][4];
        u32x4 B[2][2];
        i32x8 bf[2][4];
#define LOAD_A(slot, s, q) do { const unsigned _o = a_base + ((unsigned)(s) << 21) + (unsigned)(q) * 4096u; \
        _Pragma("unroll") for (int _u = 0; _u < 4; ++_u) A[slot][_u] = __builtin_amdgcn_raw_buffer_load_b128(rl, _o + _u * 1024u, 0, AUX_SYS); } while (0)
#define LOAD_B(slot, s, q) do { const unsigned _o = b_base + ((unsigned)(s) << 20) + (unsigned)(q) * 16384u; \
        _Pragma("unroll") for (int _u = 0; _u < 2; ++_u) B[slot][_u] = __builtin_amdgcn_raw_buffer_load_b128(rl, _o + _u * 1024u, 0, AUX_SYS); } while (0)
#define WRITE_B(slot, buf) do { _Pragma("unroll") for (int _u = 0; _u < 2; ++_u) *(u32x4*)(lw + (buf) * 16384 + _u * 1024) = B[slot][_u]; } while (0)
#define READ_B(buf, t) do { \
            _Pragma("unroll") for (int _j = 0; _j < 4; ++_j) { \
                const u32x4 b0 = *(const u32x4*)(lr + (buf) * 16384 + (2 * (t)) * 4096 + _j * 1024); \
                const u32x4 b1 = *(const u32x4*)(lr + (buf) * 16384 + (2 * (t) + 1) * 4096 + _j * 1024); \
                _Pragma("unroll") for (int _e = 0; _e < 4; ++_e) { bf[t][_j][_e] = (int)b0[_e]; bf[t][_j][4 + _e] = (int)b1[_e]; } \
            } } while (0)
#define MMA(slot, t) do { \
            i32x8 af; \
            _Pragma("unroll") for (int _e = 0; _e < 4; ++_e) { af[_e] = (int)A[slot][2 * (t)][_e]; af[4 + _e] = (int)A[slot][2 * (t) + 1][_e]; } \
            _Pragma("unroll") for (int _j = 0; _j < 4; ++_j) \
                acc[_j] = __builtin_amdgcn_mfma_scale_f32_32x32x64_f8f6f4(af, bf[t][_j], acc[_j], 1, 1, 0, 0, 0, 0); \
        } while (0)
#define SCHED() __builtin_amdgcn_sched_barrier(0)
#define CHUNK(cs, ls, lq, PRE) do { \
            READ_B((cs) & 1, 1); \
            SCHED(); \
            WRITE_B(((cs) + 1) & 1, ((cs) + 1) & 1); \
            PRE; \
            LOAD_B(((cs) + 1) & 1, ls, lq); LOAD_A(((cs) + 3) & 3, ls, lq); \
            SCHED(); \
            MMA(cs, 0); \
            SCHED(); \
            __syncthreads(); \
            READ_B(((cs) + 1) & 1, 0); \
            SCHED(); \
            MMA(cs, 1); \
            SCHED(); \
        } while (0)

        int s_cur = rank;
        ok &= wait_ge(ws, fl_off + (unsigned)s_cur * fl_str, call);
        if (round == 0) STAMP(2);
        LOAD_B(0, s_cur, 0); LOAD_A(0, s_cur, 0);
        LOAD_B(1, s_cur, 1); LOAD_A(1, s_cur, 1);
        __syncthreads();
        WRITE_B(0, 0);
        LOAD_B(0, s_cur, 2); LOAD_A(2, s_cur, 2);
        __syncthreads();
        READ_B(0, 0);
        for (int n = 0; n < 8; ++n) {
            const bool more = n < 7;
            const int s_chk = more ? (key - (n < tr ? n : n + 1)) & 7 : s_cur;
            const int s_nxt = more ? s_chk : 64;
            const unsigned fo = fl_off + (unsigned)s_chk * fl_str;
            unsigned fv;
            CHUNK(0, s_cur, 3, fv = __builtin_amdgcn_raw_buffer_load_b32(rl, fo, 0, AUX_SYS); SCHED());
            CHUNK(1, s_nxt, 0, asm volatile("" : "+v"(fv));
                               if (!__all(fv >= call)) ok &= wait_ge(ws, fo, call);
                               asm volatile("" ::: "memory");
                               if (round == 0) STAMP(3 + n));
            CHUNK(2, s_nxt, 1, );
            CHUNK(3, s_nxt, 2, );
            s_cur = s_nxt;
        }
        STAMP(11);

        if (__hip_atomic_load((unsigned*)(ws + OFF_TMO), __ATOMIC_RELAXED, __HIP_MEMORY_SCOPE_AGENT) == call) ok = false;
        const float scale = ok ? scale_ok : __builtin_nanf("");
        __syncthreads();
        if (wave == rank) {
            const unsigned o_base = ((unsigned)(ib * 32 + 4 * h) * GN + (unsigned)(rank * NC + pn * 128 + 4 * c32)) * 4u;
#pragma unroll
            for (int e = 0; e < 16; ++e) {
                u32x4 v;
#pragma unroll
                for (int j = 0; j < 4; ++j) {
                    const float y = acc[j][e] * scale;
                    v[j] = __builtin_bit_cast(unsigned, (y <= 0.f) ? 0.f : y);
                }
                const unsigned row = (unsigned)((e & 3) + 8 * (e >> 2));
                __builtin_amdgcn_raw_buffer_store_b128(v, ro, o_base + row * (GN * 4u), 0, AUX_O);
            }
        } else {
            unsigned base = 0, mlo = 0, mhi = 0;
#pragma unroll
            for (int e = 0; e < 16; ++e) {
#pragma unroll
                for (int j = 0; j < 4; ++j) {
                    const float y = acc[j][e] * scale;
                    const bool nz = !(y <= 0.f);
                    const u64 m = __ballot(nz);
                    const unsigned pos = __builtin_amdgcn_mbcnt_hi((unsigned)(m >> 32), __builtin_amdgcn_mbcnt_lo((unsigned)m, base));
                    if (nz) *(float*)(pk + 512 + pos * 4u) = y;
                    mlo = lane == e * 4 + j ? (unsigned)m : mlo;
                    mhi = lane == e * 4 + j ? (unsigned)(m >> 32) : mhi;
                    base += (unsigned)__builtin_popcountll(m);
                }
            }
            *(u32x2*)(pk + lane * 8) = u32x2{mlo, mhi};
            const unsigned nbytes = (512u + base * 4u + 63u) & ~63u;
            const rsrc_t rp = mk_rsrc(ws + OFF_PK + (unsigned)(wave * 256 + tile) * SLOT, nbytes);
            for (unsigned off = (unsigned)lane * 16u; off - (unsigned)lane * 16u < nbytes; off += 1024u) {
                const u32x4 v = *(const u32x4*)(pk + (off < SLOT ? off : 0u));
                __builtin_amdgcn_raw_buffer_store_b128(v, rp, off, 0, AUX_SYS);
            }
            const unsigned k = (ticket(ws, OFF_QP + (unsigned)wave * 128u, lane) - (call - 1u) * 256u) & 255u;
            drain();
            if (lane == 0) store_flag(p.ws[wave] + OFF_JF + (unsigned)(rank * 1024 + NITEM + k) * 64u, call, ((unsigned)tile << 16) | base);
        }
        STAMP(12);
    }

    {
        const int ord = li >= 8 ? li - 8 : li + 17;
        for (int v = (ord * 8 + xg) * 8 + wave; v < 1792 * USPLIT; v += NG * 8) {
            const int u = v / USPLIT, part = v % USPLIT;
            const int k = u / 7, src = (rank + 1 + u % 7) & 7;
            const unsigned ent = wait_flag64(ws, OFF_UF + (unsigned)(src * 256 + k) * 128u, call);
            STAMP(13);
            const unsigned cv = ent & 0xffffu;
            if (cv > 4096u) {
                if (lane == 0) ((float*)p.out)[src * NC] = __builtin_nanf("");
                continue;
            }
            const unsigned tile = (ent >> 16) & 255u;
            const rsrc_t rp = mk_rsrc(ws + OFF_RK + (unsigned)(src * 256 + k) * SLOT, (512u + cv * 4u + 63u) & ~63u);
            const u32x2 mk = __builtin_amdgcn_raw_buffer_load_b64(rp, (unsigned)lane * 8u, 0, AUX_SYS);
            u32x4 d[16];
#pragma unroll
            for (int r = 0; r < 16; ++r) d[r] = __builtin_amdgcn_raw_buffer_load_b128(rp, 512u + (unsigned)r * 1024u + (unsigned)lane * 16u, 0, AUX_SYS);
#pragma unroll
            for (int r = 0; r < 16; ++r) *(u32x4*)(pk + r * 1024 + lane * 16) = d[r];
            const unsigned o_base = ((((tile >> 4) * 32u + 4u * h) * GN) + (unsigned)(src * NC) + (tile & 15u) * 128u + 4u * c32) * 4u;
            unsigned rb = 0;
#pragma unroll
            for (int e = 0; e < 16; ++e) {
                const bool mine = USPLIT == 1 || (e / (16 / USPLIT)) == part;
                u32x4 o;
#pragma unroll
                for (int j = 0; j < 4; ++j) {
                    const unsigned lo = __builtin_amdgcn_readlane(mk[0], e * 4 + j);
                    const unsigned hi = __builtin_amdgcn_readlane(mk[1], e * 4 + j);
                    if (mine) {
                        const unsigned pos = __builtin_amdgcn_mbcnt_hi(hi, __builtin_amdgcn_mbcnt_lo(lo, rb));
                        const unsigned val = *(const unsigned*)(pk + (pos & 4095u) * 4u);
                        const bool bit = ((h ? hi : lo) >> c32) & 1u;
                        o[j] = bit ? val : 0u;
                    }
                    rb += (unsigned)__builtin_popcount(lo) + (unsigned)__builtin_popcount(hi);
                }
                const unsigned row = (unsigned)((e & 3) + 8 * (e >> 2));
                if (mine) __builtin_amdgcn_raw_buffer_store_b128(o, ro, o_base + row * (GN * 4u), 0, AUX_O);
            }
        }
    }
    STAMP(14);
}

extern "C" void kernel_launch(int rank, int nranks, ncclComm_t comm,
                              hipStream_t stream,
                              void* const* d_in, const int* in_sizes, int n_in,
                              void* d_out, int out_size,
                              const int* in_dtypes, int out_dtype,
                              void* const* d_out_all,
                              void* const* d_ws_all, size_t ws_bytes) {
    (void)comm; (void)in_sizes; (void)n_in; (void)d_out_all; (void)out_size; (void)in_dtypes; (void)out_dtype;
    if (nranks != NR || (unsigned)rank >= (unsigned)NR || ws_bytes < (size_t)WS_USED) {
        fprintf(stderr, "unsupported configuration\n");
        return;
    }
    Params p;
    p.x = (const unsigned char*)d_in[0];
    p.w = (const unsigned char*)d_in[1];
    p.sx = (const float*)d_in[2];
    p.sw = (const float*)d_in[3];
    p.out = (char*)d_out;
    for (int i = 0; i < NR; ++i) p.ws[i] = (char*)d_ws_all[i];
    p.rank = rank;
    HIP_CHECK(hipFuncSetAttribute((const void*)fused_gemm_rs, hipFuncAttributeMaxDynamicSharedMemorySize, (int)LDS_BYTES));
    hipLaunchKernelGGL(fused_gemm_rs, dim3(NWG), dim3(512), LDS_BYTES, stream, p);
    HIP_CHECK(hipGetLastError());
}
```

```cpp
#include <hip/hip_runtime.h>
#include <rccl/rccl.h>
#include <cstdio>
#include <cstdint>

#define PUSHW 4
#define FU 14
#define POLL_SLEEP 4

#define HIP_CHECK(expr) do { hipError_t _e = (expr); if (_e != hipSuccess) { \
    fprintf(stderr, "HIP error %s at %s:%d\n", hipGetErrorString(_e), __FILE__, __LINE__); \
    return; } } while (0)

typedef unsigned u32x4 __attribute__((ext_vector_type(4)));
typedef unsigned u32x2 __attribute__((ext_vector_type(2)));
typedef int i32x8 __attribute__((ext_vector_type(8)));
typedef float f32x16 __attribute__((ext_vector_type(16)));
typedef unsigned long long u64;
typedef __amdgpu_buffer_rsrc_t rsrc_t;

constexpr int NR = 8;
constexpr int GN = 16384;
constexpr int KS = 512;
constexpr int NWG = 256;
constexpr int NH = 56;
constexpr int NG = NWG - NH;
constexpr unsigned NP0 = 8 * PUSHW;
constexpr unsigned NPIECE = 256;
constexpr unsigned X_BYTES = 4096u * KS;
constexpr unsigned W_BYTES = (unsigned)KS * GN;
constexpr unsigned OUT_BYTES = 512u * GN * 4u;
constexpr unsigned SLOT = 512u + 16384u;
constexpr unsigned LSLOT = 17408u;

constexpr unsigned OFF_LX = 0;
constexpr unsigned OFF_LW = 16u << 20;
constexpr unsigned OFF_SW = 32u << 20;
constexpr unsigned OPB = 40u << 20;
constexpr unsigned OFF_FX = 2u * OPB;
constexpr unsigned OFF_FW = OFF_FX + 8u * 128u * 128u;
constexpr unsigned JOFF = 8u * 128u * 128u + 8u * 32u * 128u;
constexpr unsigned OFF_Q = OFF_FX + 2u * JOFF;
constexpr unsigned OFF_CNT = OFF_Q + 8u * 256u * 64u;
constexpr unsigned OFF_TMO = OFF_CNT + 256u * 128u;
constexpr unsigned OFF_JTK = OFF_TMO + 128u;
constexpr unsigned OFF_QP = OFF_JTK + 8u * 128u;
constexpr unsigned OFF_DBG = OFF_QP + 8u * 128u;
constexpr unsigned OFF_PK = OFF_FX + (2u << 20);
constexpr unsigned WS_USED = OFF_PK + 8u * 256u * SLOT;
static_assert(OFF_DBG + 256u * 8u * 16u * 8u <= OFF_PK, "layout");

constexpr u64 DEADLINE_TICKS = 50000000ull;
constexpr int AUX_SYS = 17;
constexpr unsigned LDS_BYTES = 8u * LSLOT + 1024u;

struct Params {
    const unsigned char* x;
    const unsigned char* w;
    const float* sx;
    const float* sw;
    char* out;
    char* ws[NR];
    int rank;
};

__device__ __forceinline__ rsrc_t mk_rsrc(const void* p, unsigned bytes) {
    u64 a = (u64)p;
    unsigned lo = __builtin_amdgcn_readfirstlane((unsigned)a);
    unsigned hi = __builtin_amdgcn_readfirstlane((unsigned)(a >> 32));
    return __builtin_amdgcn_make_buffer_rsrc((void*)(((u64)hi << 32) | lo), 0, __builtin_amdgcn_readfirstlane(bytes), 0x00020000);
}

#define SCHED() __builtin_amdgcn_sched_barrier(0)
#define STAMP(idx) do { } while (0)
#define STAMP_INIT() do { } while (0)
#define STAMP_DUMP() do { } while (0)

__device__ __forceinline__ bool wait_ge(char* ws, unsigned foff, unsigned call) {
    const unsigned* f = (const unsigned*)(ws + foff);
    unsigned* tmo = (unsigned*)(ws + OFF_TMO);
    unsigned v = __hip_atomic_load(f, __ATOMIC_RELAXED, __HIP_MEMORY_SCOPE_SYSTEM);
    if (__all(v >= call)) return true;
    const u64 t0 = __builtin_amdgcn_s_memrealtime();
    for (unsigned it = 0;; ++it) {
        __builtin_amdgcn_s_sleep(POLL_SLEEP);
        if (v < call) v = __hip_atomic_load(f, __ATOMIC_RELAXED, __HIP_MEMORY_SCOPE_SYSTEM);
        if (__all(v >= call)) return true;
        if ((it & 63u) != 63u) continue;
        const unsigned t = __builtin_amdgcn_readfirstlane(__hip_atomic_load(tmo, __ATOMIC_RELAXED, __HIP_MEMORY_SCOPE_AGENT));
        if (t == call || __builtin_amdgcn_s_memrealtime() - t0 > DEADLINE_TICKS) {
            __hip_atomic_store(tmo, call, __ATOMIC_RELAXED, __HIP_MEMORY_SCOPE_AGENT);
            return false;
        }
    }
}
__device__ __forceinline__ unsigned wait_flag64(char* ws, unsigned foff, unsigned call) {
    const u64* f = (const u64*)(ws + foff);
    unsigned* tmo = (unsigned*)(ws + OFF_TMO);
    const u64 t0 = __builtin_amdgcn_s_memrealtime();
    for (unsigned it = 0;; ++it) {
        const u64 v = __hip_atomic_load(f, __ATOMIC_RELAXED, __HIP_MEMORY_SCOPE_SYSTEM);
        const unsigned hi = __builtin_amdgcn_readfirstlane((unsigned)(v >> 32));
        const unsigned lo = __builtin_amdgcn_readfirstlane((unsigned)v);
        if (hi >= call) return lo;
        __builtin_amdgcn_s_sleep(POLL_SLEEP);
        if ((it & 63u) != 63u) continue;
        const unsigned t = __builtin_amdgcn_readfirstlane(__hip_atomic_load(tmo, __ATOMIC_RELAXED, __HIP_MEMORY_SCOPE_AGENT));
        if (t == call || __builtin_amdgcn_s_memrealtime() - t0 > DEADLINE_TICKS) {
            __hip_atomic_store(tmo, call, __ATOMIC_RELAXED, __HIP_MEMORY_SCOPE_AGENT);
            return ~0u;
        }
    }
}

__device__ __forceinline__ void drain() {
    asm volatile("" ::: "memory");
    __builtin_amdgcn_s_waitcnt(0x0F70);
    asm volatile("s_waitcnt vmcnt(0)" ::: "memory");
}
__device__ __forceinline__ void store_flag(char* addr, unsigned call, unsigned payload) {
    __hip_atomic_store((u64*)addr, ((u64)call << 32) | payload, __ATOMIC_RELAXED, __HIP_MEMORY_SCOPE_SYSTEM);
}
__device__ __forceinline__ unsigned ticket(char* ws, unsigned off, int lane) {
    unsigned r = 0;
    if (lane == 0) r = __hip_atomic_fetch_add((unsigned*)(ws + off), 1u, __ATOMIC_RELAXED, __HIP_MEMORY_SCOPE_AGENT);
    return __builtin_amdgcn_readfirstlane(r);
}

__device__ __forceinline__ unsigned other_rank(unsigned c, unsigned i) { return ((((c >> 1) + 1u + (i >> 1)) & 3u) << 1) | (i & 1u); }
__device__ __forceinline__ unsigned remote_rg(unsigned c, unsigned rho) { return other_rank(c, rho % 6u) * 16u + (c & 1u) * 8u + rho / 6u; }

__device__ __forceinline__ void load_w_item(const unsigned char* w, unsigned col0, unsigned u, int lane, unsigned (&v)[16]) {
    const rsrc_t rw = mk_rsrc(w, W_BYTES);
    const unsigned src = (32u * u + 16u * (unsigned)(lane >> 5)) * GN + col0 + 4u * (unsigned)(lane & 31);
#pragma unroll
    for (int i = 0; i < 16; ++i) v[i] = __builtin_amdgcn_raw_buffer_load_b32(rw, src + (unsigned)i * GN, 0, 0);
}
__device__ __forceinline__ void order_w_item(const unsigned (&v)[16], u32x4 (&d)[4]) {
#pragma unroll
    for (int q = 0; q < 4; ++q) {
        const unsigned lo01 = __builtin_amdgcn_perm(v[4 * q + 1], v[4 * q], 0x05010400u);
        const unsigned hi01 = __builtin_amdgcn_perm(v[4 * q + 1], v[4 * q], 0x07030602u);
        const unsigned lo23 = __builtin_amdgcn_perm(v[4 * q + 3], v[4 * q + 2], 0x05010400u);
        const unsigned hi23 = __builtin_amdgcn_perm(v[4 * q + 3], v[4 * q + 2], 0x07030602u);
        d[0][q] = __builtin_amdgcn_perm(lo23, lo01, 0x05040100u);
        d[1][q] = __builtin_amdgcn_perm(lo23, lo01, 0x07060302u);
        d[2][q] = __builtin_amdgcn_perm(hi23, hi01, 0x05040100u);
        d[3][q] = __builtin_amdgcn_perm(hi23, hi01, 0x07060302u);
    }
}
__device__ __forceinline__ void load_x_item(const unsigned char* x, unsigned rg, unsigned q, int lane, u32x4 (&d)[4]) {
    const rsrc_t rx = mk_rsrc(x, X_BYTES);
    const unsigned src = (32u * rg + (unsigned)(lane & 31)) * KS + 16u * (unsigned)(lane >> 5) + 128u * q;
#pragma unroll
    for (int r = 0; r < 4; ++r) d[r] = __builtin_amdgcn_raw_buffer_load_b128(rx, src + 32u * r, 0, 0);
}

__global__ void __launch_bounds__(512, 2) fused_gemm_rs(Params p) {
    extern __shared__ __attribute__((aligned(16))) char lds[];
    const int wave = __builtin_amdgcn_readfirstlane(threadIdx.x >> 6);
    const unsigned rank = (unsigned)p.rank;
    const int b = blockIdx.x;
    char* const ws = p.ws[rank];

    unsigned* cnt = (unsigned*)(ws + OFF_CNT + (unsigned)b * 128u + (unsigned)wave * 4u);

    if (b < NH) {
        int lane = threadIdx.x & 63;
        asm volatile("" : "+v"(lane));
        const int c32 = lane & 31, h = lane >> 5;
        STAMP_INIT();
        STAMP(0);
        const unsigned cnt_v = __hip_atomic_load(cnt, __ATOMIC_RELAXED, __HIP_MEMORY_SCOPE_AGENT);
        const unsigned peer = (rank + 1u + (unsigned)(b % 7)) & 7u;
        const bool partner = peer == (rank ^ 1u);
        char* const wsp = p.ws[peer];
        char* const wsq = p.ws[rank ^ 1u];
        const unsigned j = (unsigned)(b / 7);
        const unsigned l16 = (unsigned)lane * 16u;
        unsigned call;
        if (wave < PUSHW) {
            const unsigned m = j * PUSHW + (unsigned)wave, rg = remote_rg(peer, m >> 2), q = m & 3u;
            u32x4 d[4];
            load_x_item(p.x, rg, q, lane, d);
            call = __builtin_amdgcn_readfirstlane(cnt_v) + 1u;
            const rsrc_t rd = mk_rsrc(wsp + (call & 1u) * OPB + OFF_LX + (rank << 21) + (rg << 14) + (q << 12), 4096u);
#pragma unroll
            for (int r = 0; r < 4; ++r) __builtin_amdgcn_raw_buffer_store_b128(d[r], rd, (unsigned)r * 1024u + l16, 0, AUX_SYS);
            drain();
            if (lane == 0) store_flag(wsp + OFF_FX + (rank * 128u + rg) * 128u + q * 8u, call, 0u);
        } else {
            call = __builtin_amdgcn_readfirstlane(cnt_v) + 1u;
        }
        if (lane == 0) __hip_atomic_store(cnt, call, __ATOMIC_RELAXED, __HIP_MEMORY_SCOPE_AGENT);
        STAMP(1);

        constexpr unsigned PBUF = 24576u;
        const unsigned opb = (call & 1u) * OPB;
        constexpr unsigned XO_P = 704u + 24u * FU, XO_N = 640u + 4u * (16u - FU);
        const unsigned nij = partner ? (XO_P + 64u - NP0) / 2u : (XO_N + 64u - NP0) / 2u;
        const unsigned njob = partner ? nij : nij + NPIECE;
        const rsrc_t rl = mk_rsrc(ws, OFF_PK);
        unsigned tka = OFF_JTK + peer * 128u;
        asm volatile("" : "+v"(tka));
        const unsigned tbase = (call - 1u) * njob - 32u;
        unsigned k0 = j, k1 = j + 8u, k2 = j + 16u;
        unsigned tkv = j + 24u + tbase;
        u32x4 d[3];
        unsigned so1 = 0, do0 = 0, do1 = 0, fo0 = 0, fo1 = 0, so2 = 0, do2 = 0, fo2 = 0, jnb;
#define JOB_DESC(k, SO, DO, FO) do { \
            if ((k) < nij) { \
                const unsigned _m = NP0 + 2u * (k) + (unsigned)(wave >> 2), _sub = (unsigned)(wave & 3) * 1024u; \
                const unsigned _xo = partner ? XO_P : XO_N; \
                if (_m < 192u || _m >= _xo) { \
                    const unsigned _rg = _m < 192u ? remote_rg(rank, _m >> 2) : rank * 16u + ((_m - _xo) >> 2), _q = _m & 3u; \
                    SO = DO = opb + OFF_LX + (peer << 21) + (_rg << 14) + (_q << 12) + _sub; \
                    FO = OFF_FX + (peer * 128u + _rg) * 128u + _q * 8u; \
                } else { \
                    unsigned _pn, _s = peer, _fw = 0u, _u = _m & 15u; \
                    if (_m < 576u) _pn = ((_m - 192u) >> 7) * 8u + (((_m >> 4) + 4u + peer) & 7u); \
                    else if (!partner) { \
                        if (_m < 640u) { _pn = 24u + (rank & 1u) * 4u + (((_m >> 4) + peer) & 3u); _fw = _u < FU ? 0x80000000u : 0u; } \
                        else { _pn = 28u - (rank & 1u) * 4u + (_m & 3u); _u = FU + ((_m - 640u) >> 2); } \
                    } \
                    else if (_m < 704u) _pn = 24u + (((rank & 1u) * 4u + ((_m - 576u) >> 4)) & 7u); \
                    else { \
                        const unsigned _f = (_m - 704u) / FU; \
                        _u = (_m - 704u) % FU; _pn = 28u - (rank & 1u) * 4u + (_f & 3u); _s = other_rank(rank, _f >> 2); \
                    } \
                    const unsigned _o = (_pn << 16) + (_u << 12); \
                    DO = opb + OFF_LW + (_s << 21) + _o + _sub; \
                    SO = partner ? DO : opb + OFF_SW + ((rank >> 1) << 21) + _o + _sub; \
                    FO = (OFF_FW + _s * 4096u + (_o >> 9)) | _fw; \
                } \
            } else { \
                FO = OFF_Q - JOFF + (peer * 256u + (((k) - nij) & 255u)) * 64u; \
            } \
            SO = __builtin_amdgcn_readfirstlane(SO); DO = __builtin_amdgcn_readfirstlane(DO); FO = __builtin_amdgcn_readfirstlane(FO); \
        } while (0)
#define READY_FLAG(FO) (((FO) & 0x7fffffffu) + JOFF)
#define PIECE_BYTES(e) (((e) & 0xffffu) <= 4096u ? (512u + ((e) & 0xffffu) * 4u + 63u) & ~63u : 0u)
#define JOB_SRC(k, e, SO) do { \
            jnb = 1024u; \
            if ((k) >= nij) { \
                const unsigned _pb = PIECE_BYTES(e); \
                SO = OFF_PK + (rank * 256u + (((e) >> 16) & 255u)) * SLOT + (unsigned)wave * 1024u; \
                jnb = _pb > (unsigned)wave * 1024u ? _pb - (unsigned)wave * 1024u : 0u; \
            } \
            SO = __builtin_amdgcn_readfirstlane(SO); jnb = __builtin_amdgcn_readfirstlane(jnb); \
            asm volatile("" : "+s"(SO), "+s"(jnb));        \
        } while (0)
#define JOB_LOADS(SO) do { \
            const rsrc_t _rs = mk_rsrc(wsp + SO, jnb); \
            _Pragma("unroll") for (int _r = 0; _r < 3; ++_r) d[_r] = __builtin_amdgcn_raw_buffer_load_b128(_rs, l16 + _r * 8192u, 0, AUX_SYS); \
        } while (0)
#define JOB_NEXT(BLOCK_BELOW) do { \
            JOB_DESC(k2, so2, do2, fo2); \
            asm volatile("" : "+s"(fo2)); \
            enx = ~0u; \
            if (k1 < njob) { \
                enx = __builtin_amdgcn_readfirstlane(fv[0]); \
                if (__builtin_amdgcn_readfirstlane(fv[1]) < call) \
                    enx = k1 < (BLOCK_BELOW) ? wait_flag64(ws, READY_FLAG(fo1), call) : ~0u; \
            } \
            JOB_SRC(k1, enx, so1); \
        } while (0)
#define JOB_ADVANCE(EXTRA) do { \
            if (threadIdx.x == 0) { \
                *(unsigned*)(lds + 2 * PBUF + buf * 4u) = tkv;             \
                tkv = __hip_atomic_fetch_add((unsigned*)(ws + tka), 1u, __ATOMIC_RELAXED, __HIP_MEMORY_SCOPE_AGENT); \
                EXTRA; \
            } \
            fv = __builtin_amdgcn_raw_buffer_load_b64(rl, READY_FLAG(fo2), 0, AUX_SYS); \
            SCHED(); \
            JOB_LOADS(so1); \
            asm volatile("s_waitcnt lgkmcnt(0)" ::: "memory"); \
            __builtin_amdgcn_s_barrier(); \
            asm volatile("" ::: "memory"); \
        } while (0)
#define JOB_ROTATE() do { \
            const unsigned _tkn = *(const unsigned*)(lds + 2 * PBUF + buf * 4u); \
            ent = enx; k0 = k1; k1 = k2; do0 = do1; fo0 = fo1; so1 = so2; do1 = do2; fo1 = fo2; \
            k2 = __builtin_amdgcn_readfirstlane(_tkn) - tbase; \
            buf ^= 1u; \
        } while (0)
#define PUBLISH(FO) do { \
            if ((FO) && (threadIdx.x & 255) < 2u) { \
                if ((threadIdx.x & 1u) == 0u) store_flag(ws + ((FO) & 0x7fffffffu), call, 0u); \
                else if ((FO) & 0x80000000u) store_flag(wsq + READY_FLAG(FO), call, 0u); \
            } \
        } while (0)
        unsigned so0 = 0;
        JOB_DESC(k0, so0, do0, fo0);
        JOB_DESC(k1, so1, do1, fo1);
        unsigned ent = wait_flag64(ws, READY_FLAG(fo0), call), enx;
        u32x2 fv = __builtin_amdgcn_raw_buffer_load_b64(rl, READY_FLAG(fo1), 0, AUX_SYS);
        JOB_SRC(k0, ent, so0);
        SCHED();
        JOB_LOADS(so0);
        STAMP(2);
        unsigned buf = 0;
        unsigned pend = 0;
        int si = 3;
        while (k0 < nij) {
            JOB_NEXT(nij);
            drain();
            asm volatile("" :: "v"(d[1]), "v"(d[2]));
            __builtin_amdgcn_raw_buffer_store_b128(d[0], rl, do0 + l16, 0, AUX_SYS);
            JOB_ADVANCE(;);
            PUBLISH(pend);
            pend = fo0;
            JOB_ROTATE();
        }
        STAMP(8);
        drain();
        __builtin_amdgcn_s_barrier();
        PUBLISH(pend);
        if (k0 < njob && ent == ~0u) {
            ent = wait_flag64(ws, READY_FLAG(fo0), call);
            JOB_SRC(k0, ent, so0);
            JOB_LOADS(so0);
            drain();
        }
        int it = 0;
        const unsigned o_peer = ((peer & 1u) * 256u + 4u * (unsigned)h) * GN + (peer >> 1) * 4096u + 4u * (unsigned)c32;
        while (k0 < njob) {
            char* const pb = lds + buf * PBUF;
            const unsigned nb = PIECE_BYTES(ent);
            JOB_NEXT(njob);
            SCHED();
#pragma unroll
            for (int r = 0; r < 3; ++r) *(u32x4*)(pb + wave * 1024 + l16 + r * 8192u) = d[r];
            JOB_ADVANCE(if (nb == 0u) ((float*)p.out)[o_peer] = __builtin_nanf(""));
            const u32x2 mk = *(const u32x2*)(pb + lane * 8);
            const rsrc_t ro = mk_rsrc(p.out, nb ? OUT_BYTES : 0u);
            const unsigned pid = (ent >> 16) & 255u;
            unsigned pc = lane < wave * 8 ? (unsigned)__builtin_popcount(mk[0]) + (unsigned)__builtin_popcount(mk[1]) : 0u;
            pc += (unsigned)__builtin_amdgcn_update_dpp(0, (int)pc, 0x111, 0xf, 0xf, false);
            pc += (unsigned)__builtin_amdgcn_update_dpp(0, (int)pc, 0x112, 0xf, 0xf, false);
            pc += (unsigned)__builtin_amdgcn_update_dpp(0, (int)pc, 0x114, 0xf, 0xf, false);
            pc += (unsigned)__builtin_amdgcn_update_dpp(0, (int)pc, 0x118, 0xf, 0xf, false);
            const unsigned pre = __builtin_amdgcn_readlane(pc, 15) + __builtin_amdgcn_readlane(pc, 31) +
                                 __builtin_amdgcn_readlane(pc, 47) + __builtin_amdgcn_readlane(pc, 63);
            unsigned sb = buf * PBUF + 512u + (pre << 2);
            const unsigned o_base = (o_peer + (pid >> 5) * 32u * GN + (pid & 31u) * 128u) * 4u;
#pragma unroll
            for (int ee = 0; ee < 2; ++ee) {
                u32x4 o;
#pragma unroll
                for (int jj = 0; jj < 4; ++jj) {
                    const unsigned lo = __builtin_amdgcn_readlane(mk[0], wave * 8 + ee * 4 + jj);
                    const unsigned hi = __builtin_amdgcn_readlane(mk[1], wave * 8 + ee * 4 + jj);
                    const u64 m = ((u64)hi << 32) | lo;
                    const unsigned pos = __builtin_amdgcn_mbcnt_hi(hi, __builtin_amdgcn_mbcnt_lo(lo, 0u));
                    const unsigned val = *(const unsigned*)(lds + ((pos << 2) + sb));
                    o[jj] = __builtin_amdgcn_inverse_ballot_w64(m) ? val : 0u;
                    sb += (unsigned)__builtin_popcountll(m) << 2;
                }
                const unsigned e = (unsigned)(wave * 2 + ee);
                const unsigned row = (e & 3u) + 8u * (e >> 2);
                __builtin_amdgcn_raw_buffer_store_b128(o, ro, o_base + row * (GN * 4u), 0, AUX_SYS);
            }
            JOB_ROTATE();
            ++it;
        }
        STAMP(15);
        STAMP_DUMP();
        return;
    }

    const int g = b - NH;
    unsigned call = 0;
    float scale_ok;
    {
        const int lane = threadIdx.x & 63;
        STAMP_INIT();
        STAMP(0);
        const unsigned cnt_v = __hip_atomic_load(cnt, __ATOMIC_RELAXED, __HIP_MEMORY_SCOPE_AGENT);
        const float sxv = p.sx[0], swv = p.sw[0];
        const rsrc_t rd = mk_rsrc(ws, OFF_FX);
        const unsigned c = (unsigned)lane & 7u;
        char* const wsc = p.ws[c];
        for (unsigned jx = (unsigned)(g * 8 + wave); jx < 2560u; jx += NG * 8) {
            u32x4 d[4];
            unsigned dst, fo;
            bool need;
            if (jx < 512u) {
                const unsigned rg = (jx >> 2 & 7u) * 16u + (jx >> 5 & 1u) * 8u + (jx >> 6), q = jx & 3u;
                load_x_item(p.x, rg, q, lane, d);
                call = __builtin_amdgcn_readfirstlane(cnt_v) + 1u;
                dst = OFF_LX + (rank << 21) + (rg << 14) + (q << 12);
                fo = OFF_FX + (rank * 128u + rg) * 128u + q * 8u;
                need = c == (rg >> 4) || ((c >> 1) != (rg >> 5) && (c & 1u) == ((rg >> 3) & 1u));
            } else {
                const unsigned wi = jx - 512u, grp = wi & 3u, u = (wi >> 2) & 15u, ps = wi >> 6;
                const unsigned pn = (ps & 24u) + ((ps + rank) & 7u);
                unsigned v[16];
                load_w_item(p.w, grp * 4096u + pn * 128u, u, lane, v);
                call = __builtin_amdgcn_readfirstlane(cnt_v) + 1u;
                order_w_item(v, d);
                const unsigned o = (pn << 16) + (u << 12);
                const bool own = grp == (rank >> 1);
                dst = (own ? OFF_LW + (rank << 21) : OFF_SW + (grp << 21)) + o;
                fo = OFF_FW + rank * 4096u + (o >> 9);
                need = (c >> 1) == grp && (own || pn < 24u || u >= FU || ((pn >> 2) & 1u) == (c & 1u));
            }
            dst += (call & 1u) * OPB + (unsigned)lane * 16u;
#pragma unroll
            for (int r = 0; r < 4; ++r) __builtin_amdgcn_raw_buffer_store_b128(d[r], rd, dst + (unsigned)r * 1024u, 0, AUX_SYS);
            drain();
            if (lane < 8 && need) store_flag(wsc + fo + (c == rank ? 0u : JOFF), call, 0u);
        }
        if (lane == 0) __hip_atomic_store(cnt, call, __ATOMIC_RELAXED, __HIP_MEMORY_SCOPE_AGENT);
        scale_ok = __builtin_bit_cast(float, __builtin_amdgcn_readfirstlane(__builtin_bit_cast(unsigned, sxv * swv)));
        STAMP(1);
    }
    const rsrc_t rl = mk_rsrc(ws, OFF_PK);
    const unsigned xg = (unsigned)b & 7u, li = (unsigned)g >> 3;
    const unsigned opb = (call & 1u) * OPB;
    char* const pk = lds + wave * LSLOT;

    for (int round = 0; round < 2; ++round) {
        int ln = threadIdx.x & 63;
        asm volatile("" : "+v"(ln));
        char* const lw = lds + wave * 2048 + ln * 16;
        const char* const lr = lds + ln * 16;
        unsigned ti = li;
        if (round) { if (li >= (unsigned)(NH / 8)) break; ti = (unsigned)(NG / 8) + li; }
        const bool local = ti >= 24u;
        const unsigned pg = local ? (ti - 24u) >> 1 : ti / 6u;
        const unsigned pn = pg * 8u + (xg & 3u) * 2u + (ti & 1u);
        const unsigned rb = (local ? 3u : (ti % 6u) >> 1) * 2u + (xg >> 2);
        const unsigned rho = rb * 8u + (unsigned)wave;
        const unsigned rg = local ? rank * 16u + (rho - 48u) : remote_rg(rank, rho);
        unsigned ord = rank;
        {
            const bool fwd = pn >= 24u && ((pn >> 2) & 1u) != (rank & 1u);
            const unsigned tr = (pn - rank) & 7u;
#pragma unroll
            for (unsigned n = 1; n < 8; ++n) {
                unsigned s = (pn - (n - 1u < tr ? n - 1u : n)) & 7u;
                if (local) s = n == 7u ? rank ^ 1u : other_rank(rank, n - 1u);
                else if (fwd) s = n == 1u ? rank ^ 1u : other_rank(rank, n - 2u);
                ord |= s << (4u * n);
            }
        }

        f32x16 acc[4];
#pragma unroll
        for (int j = 0; j < 4; ++j)
#pragma unroll
            for (int e = 0; e < 16; ++e) acc[j][e] = 0.f;

        bool ok = true;
        const unsigned a_base = opb + OFF_LX + (rg << 14);
        const unsigned b_base = opb + OFF_LW + (pn << 16) + (unsigned)wave * 2048u;
        const unsigned lane16 = (unsigned)ln * 16u;
        const unsigned fl_off = (ln & 4) == 0 ? OFF_FX + rg * 128u + (unsigned)(ln & 3) * 8u + 4u
                                              : OFF_FW + pn * 128u + (unsigned)(((ln & 3) * 4 + (wave >> 1)) * 8) + 4u;
        const unsigned fl_str = (ln & 4) == 0 ? 128u * 128u : 32u * 128u;

        u32x4 A[4][4];
        u32x4 B[2][2];
        i32x8 bf[2][4];
#define LOAD_A(slot, rs, s, q) do { const unsigned _o = ((unsigned)(s) << 21) + (unsigned)(q) * 4096u + a_base; \
        _Pragma("unroll") for (int _u = 0; _u < 4; ++_u) A[slot][_u] = __builtin_amdgcn_raw_buffer_load_b128(rs, lane16, _o + _u * 1024u, AUX_SYS); } while (0)
#define LOAD_B(slot, rs, s, q) do { const unsigned _o = ((unsigned)(s) << 21) + (unsigned)(q) * 16384u + b_base; \
        _Pragma("unroll") for (int _u = 0; _u < 2; ++_u) B[slot][_u] = __builtin_amdgcn_raw_buffer_load_b128(rs, lane16, _o + _u * 1024u, AUX_SYS); } while (0)
#define WRITE_B(slot, buf) do { _Pragma("unroll") for (int _u = 0; _u < 2; ++_u) *(u32x4*)(lw + (buf) * 16384 + _u * 1024) = B[slot][_u]; } while (0)
#define READ_B(buf, t) do { \
            _Pragma("unroll") for (int _j = 0; _j < 4; ++_j) { \
                const u32x4 b0 = *(const u32x4*)(lr + (buf) * 16384 + (2 * (t)) * 4096 + _j * 1024); \
                const u32x4 b1 = *(const u32x4*)(lr + (buf) * 16384 + (2 * (t) + 1) * 4096 + _j * 1024); \
                _Pragma("unroll") for (int _e = 0; _e < 4; ++_e) { bf[t][_j][_e] = (int)b0[_e]; bf[t][_j][4 + _e] = (int)b1[_e]; } \
            } } while (0)
#define MMA(slot, t) do { \
            i32x8 af; \
            _Pragma("unroll") for (int _e = 0; _e < 4; ++_e) { af[_e] = (int)A[slot][2 * (t)][_e]; af[4 + _e] = (int)A[slot][2 * (t) + 1][_e]; } \
            _Pragma("unroll") for (int _j = 0; _j < 4; ++_j) \
                acc[_j] = __builtin_amdgcn_mfma_scale_f32_32x32x64_f8f6f4(af, bf[t][_j], acc[_j], 1, 1, 0, 0, 0, 0); \
        } while (0)
#define CHUNK(cs, rs, ls, lq, PRE) do { \
            READ_B((cs) & 1, 1); \
            SCHED(); \
            WRITE_B(((cs) + 1) & 1, ((cs) + 1) & 1); \
            SCHED(); \
            PRE; \
            LOAD_B(((cs) + 1) & 1, rs, ls, lq); LOAD_A(((cs) + 3) & 3, rs, ls, lq); \
            SCHED(); \
            MMA(cs, 0); \
            SCHED(); \
            __syncthreads(); \
            READ_B(((cs) + 1) & 1, 0); \
            SCHED(); \
            MMA(cs, 1); \
            SCHED(); \
        } while (0)

        unsigned s_cur = rank;
        ok &= wait_ge(ws, fl_off + s_cur * fl_str, call);
        if (round == 0) STAMP(2);
        LOAD_B(0, rl, s_cur, 0); LOAD_A(0, rl, s_cur, 0);
        LOAD_B(1, rl, s_cur, 1); LOAD_A(1, rl, s_cur, 1);
        __syncthreads();
        WRITE_B(0, 0);
        LOAD_B(0, rl, s_cur, 2); LOAD_A(2, rl, s_cur, 2);
        __syncthreads();
        READ_B(0, 0);
        for (int n = 0; n < 8; ++n) {
            const bool more = n < 7;
            ord >>= 4;
            const unsigned s_nxt = ord & 7u;
            const rsrc_t rn = mk_rsrc(ws, more ? OFF_PK : 0u);
            const unsigned fo = more ? fl_off + s_nxt * fl_str : OFF_TMO;
            unsigned fv;
            CHUNK(0, rl, s_cur, 3, fv = __builtin_amdgcn_raw_buffer_load_b32(rl, fo, 0, AUX_SYS); SCHED());
            CHUNK(1, rn, s_nxt, 0, asm volatile("" : "+v"(fv));
                               if (!more) ok &= !__any(fv == call);
                               else if (!__all(fv >= call)) ok &= wait_ge(ws, fo, call);
                               asm volatile("" ::: "memory");
                               if (round == 0) STAMP(3 + n));
            CHUNK(2, rn, s_nxt, 1, );
            CHUNK(3, rn, s_nxt, 2, );
            s_cur = s_nxt;
        }
        STAMP(11 + 2 * round);

        const float scale = ok ? scale_ok : __builtin_nanf("");
        int tz = threadIdx.x & 63;
        asm volatile("" : "+v"(tz));
        if (local) {
            const rsrc_t ro = mk_rsrc(p.out, OUT_BYTES);
            const unsigned o_base = (((rg & 15u) * 32u + 4u * (unsigned)(tz >> 5)) * GN + (rank >> 1) * 4096u + pn * 128u + 4u * (unsigned)(tz & 31)) * 4u;
#pragma unroll
            for (int e = 0; e < 16; ++e) {
                u32x4 v;
#pragma unroll
                for (int j = 0; j < 4; ++j) {
                    const float y = acc[j][e] * scale;
                    v[j] = __builtin_bit_cast(unsigned, (y <= 0.f) ? 0.f : y);
                }
                const unsigned row = (unsigned)((e & 3) + 8 * (e >> 2));
                __builtin_amdgcn_raw_buffer_store_b128(v, ro, o_base + row * (GN * 4u), 0, AUX_SYS);
            }
        } else {
            const unsigned owner = rg >> 4, pid = (rg & 7u) * 32u + pn;
            unsigned tk = OFF_QP + owner * 128u;
            asm volatile("" : "+v"(tk));
            unsigned kq = 0;
            if (tz == 0) kq = __hip_atomic_fetch_add((unsigned*)(ws + tk), 1u, __ATOMIC_RELAXED, __HIP_MEMORY_SCOPE_AGENT);
            asm volatile("s_waitcnt lgkmcnt(0)" ::: "memory");
            __builtin_amdgcn_s_barrier();
            asm volatile("" ::: "memory");
            const unsigned sb0 = (unsigned)(wave * LSLOT) + 512u;
            const unsigned dummy = (unsigned)(wave * LSLOT) + (unsigned)tz * 4u;
            unsigned sb = sb0, mlo = 0, mhi = 0;
#pragma unroll
            for (int e = 0; e < 16; ++e) {
#pragma unroll
                for (int j = 0; j < 4; ++j) {
                    const float y = acc[j][e] * scale;
                    const bool nz = !(y <= 0.f);
                    const u64 m = __ballot(nz);
                    const unsigned pos = __builtin_amdgcn_mbcnt_hi((unsigned)(m >> 32), __builtin_amdgcn_mbcnt_lo((unsigned)m, 0u));
                    unsigned ad = (pos << 2) + sb;
                    asm("" : "+v"(ad));
                    *(__attribute__((address_space(3))) float*)(size_t)(nz ? ad : dummy) = y;
                    asm("s_nop 1\n\tv_writelane_b32 %0, %2, %4\n\tv_writelane_b32 %1, %3, %4"
                        : "+v"(mlo), "+v"(mhi) : "s"((unsigned)m), "s"((unsigned)(m >> 32)), "i"(e * 4 + j));
                    sb += (unsigned)__builtin_popcountll(m) << 2;
                }
            }
            *(u32x2*)(pk + tz * 8) = u32x2{mlo, mhi};
            const unsigned nbytes = (512u + (sb - sb0) + 63u) & ~63u;
            const rsrc_t rp = mk_rsrc(ws + OFF_PK + (owner * 256u + pid) * SLOT, nbytes);
            for (unsigned off = 0; off < nbytes; off += 1024u) {
                const u32x4 v = *(const u32x4*)(pk + off + tz * 16);
                __builtin_amdgcn_raw_buffer_store_b128(v, rp, (unsigned)tz * 16u + off, 0, AUX_SYS);
            }
            drain();
            asm volatile("" : "+v"(kq));
            if (tz == 0) store_flag(p.ws[owner] + OFF_Q + (rank * 256u + (kq & 255u)) * 64u, call, (pid << 16) | ((sb - sb0) >> 2));
        }
        STAMP(12 + 2 * round);
    }
    STAMP_DUMP();
}

extern "C" void kernel_launch(int rank, int nranks, ncclComm_t comm,
                              hipStream_t stream,
                              void* const* d_in, const int* in_sizes, int n_in,
                              void* d_out, int out_size,
                              const int* in_dtypes, int out_dtype,
                              void* const* d_out_all,
                              void* const* d_ws_all, size_t ws_bytes) {
    (void)comm; (void)in_sizes; (void)n_in; (void)d_out_all; (void)out_size; (void)in_dtypes; (void)out_dtype;
    if (nranks != NR || (unsigned)rank >= (unsigned)NR || ws_bytes < (size_t)WS_USED) {
        fprintf(stderr, "unsupported configuration\n");
        return;
    }
    Params p;
    p.x = (const unsigned char*)d_in[0];
    p.w = (const unsigned char*)d_in[1];
    p.sx = (const float*)d_in[2];
    p.sw = (const float*)d_in[3];
    p.out = (char*)d_out;
    for (int i = 0; i < NR; ++i) p.ws[i] = (char*)d_ws_all[i];
    p.rank = rank;
    HIP_CHECK(hipFuncSetAttribute((const void*)fused_gemm_rs, hipFuncAttributeMaxDynamicSharedMemorySize, (int)LDS_BYTES));
    hipLaunchKernelGGL(fused_gemm_rs, dim3(NWG), dim3(512), LDS_BYTES, stream, p);
    HIP_CHECK(hipGetLastError());
}
```

```cpp
#include <hip/hip_runtime.h>
#include <rccl/rccl.h>
#include <cstdio>
#include <cstdint>

#define HIP_CHECK(expr) do { hipError_t _e = (expr); if (_e != hipSuccess) { \
    fprintf(stderr, "HIP error %s at %s:%d\n", hipGetErrorString(_e), __FILE__, __LINE__); \
    return; } } while (0)

typedef unsigned u32x4 __attribute__((ext_vector_type(4)));
typedef unsigned u32x2 __attribute__((ext_vector_type(2)));
typedef int i32x8 __attribute__((ext_vector_type(8)));
typedef float f32x16 __attribute__((ext_vector_type(16)));
typedef unsigned long long u64;
typedef __amdgpu_buffer_rsrc_t rsrc_t;

constexpr int NR = 8;
constexpr int GN = 16384;
constexpr int KS = 512;
constexpr int NWG = 256;
constexpr int NH = 56;
constexpr int NG = NWG - NH;
constexpr int PUSHW = 4;
constexpr unsigned NP0 = 8 * PUSHW;
constexpr unsigned FU = 14;
constexpr int POLL_SLEEP = 4;
constexpr unsigned NPIECE = 256;
constexpr unsigned X_BYTES = 4096u * KS;
constexpr unsigned W_BYTES = (unsigned)KS * GN;
constexpr unsigned OUT_BYTES = 512u * GN * 4u;
constexpr unsigned SLOT = 512u + 16384u;
constexpr unsigned LSLOT = 17408u;

constexpr unsigned OFF_LX = 0;
constexpr unsigned OFF_LW = 16u << 20;
constexpr unsigned OFF_SW = 32u << 20;
constexpr unsigned OPB = 40u << 20;
constexpr unsigned OFF_FX = 2u * OPB;
constexpr unsigned OFF_FW = OFF_FX + 8u * 128u * 128u;
constexpr unsigned JOFF = 8u * 128u * 128u + 8u * 32u * 128u;
constexpr unsigned OFF_Q = OFF_FX + 2u * JOFF;
constexpr unsigned OFF_CNT = OFF_Q + 8u * 256u * 64u;
constexpr unsigned OFF_TMO = OFF_CNT + 256u * 128u;
constexpr unsigned OFF_JTK = OFF_TMO + 128u;
constexpr unsigned OFF_QP = OFF_JTK + 8u * 128u;
constexpr unsigned OFF_PK = OFF_FX + (2u << 20);
constexpr unsigned WS_USED = OFF_PK + 8u * 256u * SLOT;
static_assert(OFF_QP + 8u * 128u <= OFF_PK, "layout");

constexpr u64 DEADLINE_TICKS = 50000000ull;
constexpr int AUX_SYS = 17;
constexpr unsigned LDS_BYTES = 8u * LSLOT;

struct Params {
    const unsigned char* x;
    const unsigned char* w;
    const float* sx;
    const float* sw;
    char* out;
    char* ws[NR];
    int rank;
};

__device__ __forceinline__ rsrc_t mk_rsrc(const void* p, unsigned bytes) {
    u64 a = (u64)p;
    unsigned lo = __builtin_amdgcn_readfirstlane((unsigned)a);
    unsigned hi = __builtin_amdgcn_readfirstlane((unsigned)(a >> 32));
    return __builtin_amdgcn_make_buffer_rsrc((void*)(((u64)hi << 32) | lo), 0, __builtin_amdgcn_readfirstlane(bytes), 0x00020000);
}

#define SCHED() __builtin_amdgcn_sched_barrier(0)

__device__ __forceinline__ bool wait_ge(char* ws, unsigned foff, unsigned call) {
    const unsigned* f = (const unsigned*)(ws + foff);
    unsigned* tmo = (unsigned*)(ws + OFF_TMO);
    unsigned v = __hip_atomic_load(f, __ATOMIC_RELAXED, __HIP_MEMORY_SCOPE_SYSTEM);
    if (__all(v >= call)) return true;
    const u64 t0 = __builtin_amdgcn_s_memrealtime();
    for (unsigned it = 0;; ++it) {
        __builtin_amdgcn_s_sleep(POLL_SLEEP);
        if (v < call) v = __hip_atomic_load(f, __ATOMIC_RELAXED, __HIP_MEMORY_SCOPE_SYSTEM);
        if (__all(v >= call)) return true;
        if ((it & 63u) != 63u) continue;
        const unsigned t = __builtin_amdgcn_readfirstlane(__hip_atomic_load(tmo, __ATOMIC_RELAXED, __HIP_MEMORY_SCOPE_AGENT));
        if (t == call || __builtin_amdgcn_s_memrealtime() - t0 > DEADLINE_TICKS) {
            __hip_atomic_store(tmo, call, __ATOMIC_RELAXED, __HIP_MEMORY_SCOPE_AGENT);
            return false;
        }
    }
}
__device__ __forceinline__ unsigned wait_flag64(char* ws, unsigned foff, unsigned call) {
    const u64* f = (const u64*)(ws + foff);
    unsigned* tmo = (unsigned*)(ws + OFF_TMO);
    const u64 t0 = __builtin_amdgcn_s_memrealtime();
    for (unsigned it = 0;; ++it) {
        const u64 v = __hip_atomic_load(f, __ATOMIC_RELAXED, __HIP_MEMORY_SCOPE_SYSTEM);
        const unsigned hi = __builtin_amdgcn_readfirstlane((unsigned)(v >> 32));
        const unsigned lo = __builtin_amdgcn_readfirstlane((unsigned)v);
        if (hi >= call) return lo;
        __builtin_amdgcn_s_sleep(POLL_SLEEP);
        if ((it & 63u) != 63u) continue;
        const unsigned t = __builtin_amdgcn_readfirstlane(__hip_atomic_load(tmo, __ATOMIC_RELAXED, __HIP_MEMORY_SCOPE_AGENT));
        if (t == call || __builtin_amdgcn_s_memrealtime() - t0 > DEADLINE_TICKS) {
            __hip_atomic_store(tmo, call, __ATOMIC_RELAXED, __HIP_MEMORY_SCOPE_AGENT);
            return ~0u;
        }
    }
}

__device__ __forceinline__ void drain() {
    asm volatile("" ::: "memory");
    __builtin_amdgcn_s_waitcnt(0x0F70);
    asm volatile("s_waitcnt vmcnt(0)" ::: "memory");
}
__device__ __forceinline__ void store_flag(char* addr, unsigned call, unsigned payload) {
    __hip_atomic_store((u64*)addr, ((u64)call << 32) | payload, __ATOMIC_RELAXED, __HIP_MEMORY_SCOPE_SYSTEM);
}

__device__ __forceinline__ unsigned other_rank(unsigned c, unsigned i) { return ((((c >> 1) + 1u + (i >> 1)) & 3u) << 1) | (i & 1u); }
__device__ __forceinline__ unsigned remote_rg(unsigned c, unsigned rho) { return other_rank(c, rho % 6u) * 16u + (c & 1u) * 8u + rho / 6u; }

__device__ __forceinline__ void load_w_item(const unsigned char* w, unsigned col0, unsigned u, int lane, unsigned (&v)[16]) {
    const rsrc_t rw = mk_rsrc(w, W_BYTES);
    const unsigned src = (32u * u + 16u * (unsigned)(lane >> 5)) * GN + col0 + 4u * (unsigned)(lane & 31);
#pragma unroll
    for (int i = 0; i < 16; ++i) v[i] = __builtin_amdgcn_raw_buffer_load_b32(rw, src + (unsigned)i * GN, 0, 0);
}
__device__ __forceinline__ void order_w_item(const unsigned (&v)[16], u32x4 (&d)[4]) {
#pragma unroll
    for (int q = 0; q < 4; ++q) {
        const unsigned lo01 = __builtin_amdgcn_perm(v[4 * q + 1], v[4 * q], 0x05010400u);
        const unsigned hi01 = __builtin_amdgcn_perm(v[4 * q + 1], v[4 * q], 0x07030602u);
        const unsigned lo23 = __builtin_amdgcn_perm(v[4 * q + 3], v[4 * q + 2], 0x05010400u);
        const unsigned hi23 = __builtin_amdgcn_perm(v[4 * q + 3], v[4 * q + 2], 0x07030602u);
        d[0][q] = __builtin_amdgcn_perm(lo23, lo01, 0x05040100u);
        d[1][q] = __builtin_amdgcn_perm(lo23, lo01, 0x07060302u);
        d[2][q] = __builtin_amdgcn_perm(hi23, hi01, 0x05040100u);
        d[3][q] = __builtin_amdgcn_perm(hi23, hi01, 0x07060302u);
    }
}
__device__ __forceinline__ void load_x_item(const unsigned char* x, unsigned rg, unsigned q, int lane, u32x4 (&d)[4]) {
    const rsrc_t rx = mk_rsrc(x, X_BYTES);
    const unsigned src = (32u * rg + (unsigned)(lane & 31)) * KS + 16u * (unsigned)(lane >> 5) + 128u * q;
#pragma unroll
    for (int r = 0; r < 4; ++r) d[r] = __builtin_amdgcn_raw_buffer_load_b128(rx, src + 32u * r, 0, 0);
}

__global__ void __launch_bounds__(512, 2) fused_gemm_rs(Params p) {
    extern __shared__ __attribute__((aligned(16))) char lds[];
    const int wave = __builtin_amdgcn_readfirstlane(threadIdx.x >> 6);
    const unsigned rank = (unsigned)p.rank;
    const int b = blockIdx.x;
    char* const ws = p.ws[rank];

    unsigned* cnt = (unsigned*)(ws + OFF_CNT + (unsigned)b * 128u + (unsigned)wave * 4u);

    if (b < NH) {
        int lane = threadIdx.x & 63;
        asm volatile("" : "+v"(lane));
        const int c32 = lane & 31, h = lane >> 5;
        const unsigned cnt_v = __hip_atomic_load(cnt, __ATOMIC_RELAXED, __HIP_MEMORY_SCOPE_AGENT);
        const unsigned peer = (rank + 1u + (unsigned)(b % 7)) & 7u;
        const bool partner = peer == (rank ^ 1u);
        char* const wsp = p.ws[peer];
        char* const wsq = p.ws[rank ^ 1u];
        const unsigned j = (unsigned)(b / 7);
        const unsigned l16 = (unsigned)lane * 16u;
        unsigned call;
        if (wave < PUSHW) {
            const unsigned m = j * PUSHW + (unsigned)wave, rg = remote_rg(peer, m >> 2), q = m & 3u;
            u32x4 d[4];
            load_x_item(p.x, rg, q, lane, d);
            call = __builtin_amdgcn_readfirstlane(cnt_v) + 1u;
            const rsrc_t rd = mk_rsrc(wsp + (call & 1u) * OPB + OFF_LX + (rank << 21) + (rg << 14) + (q << 12), 4096u);
#pragma unroll
            for (int r = 0; r < 4; ++r) __builtin_amdgcn_raw_buffer_store_b128(d[r], rd, (unsigned)r * 1024u + l16, 0, AUX_SYS);
            drain();
            if (lane == 0) store_flag(wsp + OFF_FX + (rank * 128u + rg) * 128u + q * 8u, call, 0u);
        } else {
            call = __builtin_amdgcn_readfirstlane(cnt_v) + 1u;
        }
        if (lane == 0) __hip_atomic_store(cnt, call, __ATOMIC_RELAXED, __HIP_MEMORY_SCOPE_AGENT);

        constexpr unsigned PBUF = 24576u;
        const unsigned opb = (call & 1u) * OPB;
        constexpr unsigned XO_P = 704u + 24u * FU, XO_N = 640u + 4u * (16u - FU);
        const unsigned nij = partner ? (XO_P + 64u - NP0) / 2u : (XO_N + 64u - NP0) / 2u;
        const unsigned njob = partner ? nij : nij + NPIECE;
        const rsrc_t rl = mk_rsrc(ws, OFF_PK);
        unsigned tka = OFF_JTK + peer * 128u;
        asm volatile("" : "+v"(tka));
        const unsigned tbase = (call - 1u) * njob - 32u;
        unsigned k0 = j, k1 = j + 8u, k2 = j + 16u;
        unsigned tkv = j + 24u + tbase;
        u32x4 d[3];
        unsigned so1 = 0, do0 = 0, do1 = 0, fo0 = 0, fo1 = 0, so2 = 0, do2 = 0, fo2 = 0, jnb;
#define JOB_DESC(k, SO, DO, FO) do { \
            if ((k) < nij) { \
                const unsigned _m = NP0 + 2u * (k) + (unsigned)(wave >> 2), _sub = (unsigned)(wave & 3) * 1024u; \
                const unsigned _xo = partner ? XO_P : XO_N; \
                if (_m < 192u || _m >= _xo) { \
                    const unsigned _rg = _m < 192u ? remote_rg(rank, _m >> 2) : rank * 16u + ((_m - _xo) >> 2), _q = _m & 3u; \
                    SO = DO = opb + OFF_LX + (peer << 21) + (_rg << 14) + (_q << 12) + _sub; \
                    FO = OFF_FX + (peer * 128u + _rg) * 128u + _q * 8u; \
                } else { \
                    unsigned _pn, _s = peer, _fw = 0u, _u = _m & 15u; \
                    if (_m < 576u) _pn = ((_m - 192u) >> 7) * 8u + (((_m >> 4) + 4u + peer) & 7u); \
                    else if (!partner) { \
                        if (_m < 640u) { _pn = 24u + (rank & 1u) * 4u + (((_m >> 4) + peer) & 3u); _fw = _u < FU ? 0x80000000u : 0u; } \
                        else { _pn = 28u - (rank & 1u) * 4u + (_m & 3u); _u = FU + ((_m - 640u) >> 2); } \
                    } \
                    else if (_m < 704u) _pn = 24u + (((rank & 1u) * 4u + ((_m - 576u) >> 4)) & 7u); \
                    else { \
                        const unsigned _f = (_m - 704u) / FU; \
                        _u = (_m - 704u) % FU; _pn = 28u - (rank & 1u) * 4u + (_f & 3u); _s = other_rank(rank, _f >> 2); \
                    } \
                    const unsigned _o = (_pn << 16) + (_u << 12); \
                    DO = opb + OFF_LW + (_s << 21) + _o + _sub; \
                    SO = partner ? DO : opb + OFF_SW + ((rank >> 1) << 21) + _o + _sub; \
                    FO = (OFF_FW + _s * 4096u + (_o >> 9)) | _fw; \
                } \
            } else { \
                FO = OFF_Q - JOFF + (peer * 256u + (((k) - nij) & 255u)) * 64u; \
            } \
            SO = __builtin_amdgcn_readfirstlane(SO); DO = __builtin_amdgcn_readfirstlane(DO); FO = __builtin_amdgcn_readfirstlane(FO); \
        } while (0)
#define READY_FLAG(FO) (((FO) & 0x7fffffffu) + JOFF)
#define PIECE_BYTES(e) (((e) & 0xffffu) <= 4096u ? (512u + ((e) & 0xffffu) * 4u + 63u) & ~63u : 0u)
#define JOB_SRC(k, e, SO) do { \
            jnb = 1024u; \
            if ((k) >= nij) { \
                const unsigned _pb = PIECE_BYTES(e); \
                SO = OFF_PK + (rank * 256u + (((e) >> 16) & 255u)) * SLOT + (unsigned)wave * 1024u; \
                jnb = _pb > (unsigned)wave * 1024u ? _pb - (unsigned)wave * 1024u : 0u; \
            } \
            SO = __builtin_amdgcn_readfirstlane(SO); jnb = __builtin_amdgcn_readfirstlane(jnb); \
            asm volatile("" : "+s"(SO), "+s"(jnb));        \
        } while (0)
#define JOB_LOADS(SO) do { \
            const rsrc_t _rs = mk_rsrc(wsp + SO, jnb); \
            _Pragma("unroll") for (int _r = 0; _r < 3; ++_r) d[_r] = __builtin_amdgcn_raw_buffer_load_b128(_rs, l16 + _r * 8192u, 0, AUX_SYS); \
        } while (0)
#define JOB_NEXT(BLOCK_BELOW) do { \
            JOB_DESC(k2, so2, do2, fo2); \
            asm volatile("" : "+s"(fo2)); \
            enx = ~0u; \
            if (k1 < njob) { \
                enx = __builtin_amdgcn_readfirstlane(fv[0]); \
                if (__builtin_amdgcn_readfirstlane(fv[1]) < call) \
                    enx = k1 < (BLOCK_BELOW) ? wait_flag64(ws, READY_FLAG(fo1), call) : ~0u; \
            } \
            JOB_SRC(k1, enx, so1); \
        } while (0)
#define JOB_ADVANCE() do { \
            if (threadIdx.x == 0) { \
                *(unsigned*)(lds + 2 * PBUF + buf * 4u) = tkv;             \
                tkv = __hip_atomic_fetch_add((unsigned*)(ws + tka), 1u, __ATOMIC_RELAXED, __HIP_MEMORY_SCOPE_AGENT); \
            } \
            fv = __builtin_amdgcn_raw_buffer_load_b64(rl, READY_FLAG(fo2), 0, AUX_SYS); \
            SCHED(); \
            JOB_LOADS(so1); \
            asm volatile("s_waitcnt lgkmcnt(0)" ::: "memory"); \
            __builtin_amdgcn_s_barrier(); \
            asm volatile("" ::: "memory"); \
        } while (0)
#define JOB_ROTATE() do { \
            const unsigned _tkn = *(const unsigned*)(lds + 2 * PBUF + buf * 4u); \
            ent = enx; k0 = k1; k1 = k2; do0 = do1; fo0 = fo1; so1 = so2; do1 = do2; fo1 = fo2; \
            k2 = __builtin_amdgcn_readfirstlane(_tkn) - tbase; \
            buf ^= 1u; \
        } while (0)
#define PUBLISH(FO) do { \
            if ((FO) && (threadIdx.x & 255) < 2u) { \
                if ((threadIdx.x & 1u) == 0u) store_flag(ws + ((FO) & 0x7fffffffu), call, 0u); \
                else if ((FO) & 0x80000000u) store_flag(wsq + READY_FLAG(FO), call, 0u); \
            } \
        } while (0)
        unsigned so0 = 0;
        JOB_DESC(k0, so0, do0, fo0);
        JOB_DESC(k1, so1, do1, fo1);
        unsigned ent = wait_flag64(ws, READY_FLAG(fo0), call), enx;
        u32x2 fv = __builtin_amdgcn_raw_buffer_load_b64(rl, READY_FLAG(fo1), 0, AUX_SYS);
        JOB_SRC(k0, ent, so0);
        SCHED();
        JOB_LOADS(so0);
        unsigned buf = 0;
        unsigned pend = 0;
        while (k0 < nij) {
            JOB_NEXT(nij);
            drain();
            asm volatile("" :: "v"(d[1]), "v"(d[2]));
            __builtin_amdgcn_raw_buffer_store_b128(d[0], rl, do0 + l16, 0, AUX_SYS);
            JOB_ADVANCE();
            PUBLISH(pend);
            pend = fo0;
            JOB_ROTATE();
        }
        drain();
        __builtin_amdgcn_s_barrier();
        PUBLISH(pend);
        if (k0 < njob && ent == ~0u) {
            ent = wait_flag64(ws, READY_FLAG(fo0), call);
            JOB_SRC(k0, ent, so0);
            JOB_LOADS(so0);
            drain();
        }
        const unsigned o_peer = ((peer & 1u) * 256u + 4u * (unsigned)h) * GN + (peer >> 1) * 4096u + 4u * (unsigned)c32;
        unsigned lastp = 0, bad = 0;
        while (k0 < njob) {
            char* const pb = lds + buf * PBUF;
            const unsigned nb = PIECE_BYTES(ent);
            JOB_NEXT(njob);
            SCHED();
#pragma unroll
            for (int r = 0; r < 3; ++r) *(u32x4*)(pb + wave * 1024 + l16 + r * 8192u) = d[r];
            JOB_ADVANCE();
            const u32x2 mk = *(const u32x2*)(pb + lane * 8);
            const rsrc_t ro = mk_rsrc(p.out, nb ? OUT_BYTES : 0u);
            const unsigned pid = (ent >> 16) & 255u;
            if (nb) lastp = pid; else bad = 1u;
            unsigned pc = lane < wave * 8 ? (unsigned)__builtin_popcount(mk[0]) + (unsigned)__builtin_popcount(mk[1]) : 0u;
            pc += (unsigned)__builtin_amdgcn_update_dpp(0, (int)pc, 0x111, 0xf, 0xf, false);
            pc += (unsigned)__builtin_amdgcn_update_dpp(0, (int)pc, 0x112, 0xf, 0xf, false);
            pc += (unsigned)__builtin_amdgcn_update_dpp(0, (int)pc, 0x114, 0xf, 0xf, false);
            pc += (unsigned)__builtin_amdgcn_update_dpp(0, (int)pc, 0x118, 0xf, 0xf, false);
            const unsigned pre = __builtin_amdgcn_readlane(pc, 15) + __builtin_amdgcn_readlane(pc, 31) +
                                 __builtin_amdgcn_readlane(pc, 47) + __builtin_amdgcn_readlane(pc, 63);
            unsigned sb = buf * PBUF + 512u + (pre << 2);
            const unsigned o_base = (o_peer + (pid >> 5) * 32u * GN + (pid & 31u) * 128u) * 4u;
#pragma unroll
            for (int ee = 0; ee < 2; ++ee) {
                u32x4 o;
#pragma unroll
                for (int jj = 0; jj < 4; ++jj) {
                    const unsigned lo = __builtin_amdgcn_readlane(mk[0], wave * 8 + ee * 4 + jj);
                    const unsigned hi = __builtin_amdgcn_readlane(mk[1], wave * 8 + ee * 4 + jj);
                    const u64 m = ((u64)hi << 32) | lo;
                    const unsigned pos = __builtin_amdgcn_mbcnt_hi(hi, __builtin_amdgcn_mbcnt_lo(lo, 0u));
                    const unsigned val = *(const unsigned*)(lds + ((pos << 2) + sb));
                    o[jj] = __builtin_amdgcn_inverse_ballot_w64(m) ? val : 0u;
                    sb += (unsigned)__builtin_popcountll(m) << 2;
                }
                const unsigned e = (unsigned)(wave * 2 + ee);
                const unsigned row = (e & 3u) + 8u * (e >> 2);
                __builtin_amdgcn_raw_buffer_store_b128(o, ro, o_base + row * (GN * 4u), 0, AUX_SYS);
            }
            JOB_ROTATE();
        }
        if (bad) {
            drain();
            const unsigned e = (unsigned)wave * 2u;
            if (lane == 0) ((float*)p.out)[o_peer + ((lastp >> 5) * 32u + (e & 3u) + 8u * (e >> 2)) * GN + (lastp & 31u) * 128u] = __builtin_nanf("");
        }
        return;
    }

    const int g = b - NH;
    unsigned call = 0;
    float scale_ok;
    {
        const int lane = threadIdx.x & 63;
        const unsigned cnt_v = __hip_atomic_load(cnt, __ATOMIC_RELAXED, __HIP_MEMORY_SCOPE_AGENT);
        const float sxv = p.sx[0], swv = p.sw[0];
        const rsrc_t rd = mk_rsrc(ws, OFF_FX);
        const unsigned c = (unsigned)lane & 7u;
        char* const wsc = p.ws[c];
        for (unsigned jx = (unsigned)(g * 8 + wave); jx < 2560u; jx += NG * 8) {
            u32x4 d[4];
            unsigned dst, fo;
            bool need;
            if (jx < 512u) {
                const unsigned rg = (jx >> 2 & 7u) * 16u + (jx >> 5 & 1u) * 8u + (jx >> 6), q = jx & 3u;
                load_x_item(p.x, rg, q, lane, d);
                call = __builtin_amdgcn_readfirstlane(cnt_v) + 1u;
                dst = OFF_LX + (rank << 21) + (rg << 14) + (q << 12);
                fo = OFF_FX + (rank * 128u + rg) * 128u + q * 8u;
                need = c == (rg >> 4) || ((c >> 1) != (rg >> 5) && (c & 1u) == ((rg >> 3) & 1u));
            } else {
                const unsigned wi = jx - 512u, grp = wi & 3u, u = (wi >> 2) & 15u, ps = wi >> 6;
                const unsigned pn = (ps & 24u) + ((ps + rank) & 7u);
                unsigned v[16];
                load_w_item(p.w, grp * 4096u + pn * 128u, u, lane, v);
                call = __builtin_amdgcn_readfirstlane(cnt_v) + 1u;
                order_w_item(v, d);
                const unsigned o = (pn << 16) + (u << 12);
                const bool own = grp == (rank >> 1);
                dst = (own ? OFF_LW + (rank << 21) : OFF_SW + (grp << 21)) + o;
                fo = OFF_FW + rank * 4096u + (o >> 9);
                need = (c >> 1) == grp && (own || pn < 24u || u >= FU || ((pn >> 2) & 1u) == (c & 1u));
            }
            dst += (call & 1u) * OPB + (unsigned)lane * 16u;
#pragma unroll
            for (int r = 0; r < 4; ++r) __builtin_amdgcn_raw_buffer_store_b128(d[r], rd, dst + (unsigned)r * 1024u, 0, AUX_SYS);
            drain();
            if (lane < 8 && need) store_flag(wsc + fo + (c == rank ? 0u : JOFF), call, 0u);
        }
        if (lane == 0) __hip_atomic_store(cnt, call, __ATOMIC_RELAXED, __HIP_MEMORY_SCOPE_AGENT);
        scale_ok = __builtin_bit_cast(float, __builtin_amdgcn_readfirstlane(__builtin_bit_cast(unsigned, sxv * swv)));
    }
    const rsrc_t rl = mk_rsrc(ws, OFF_PK);
    const unsigned xg = (unsigned)b & 7u, li = (unsigned)g >> 3;
    const unsigned opb = (call & 1u) * OPB;
    char* const pk = lds + wave * LSLOT;

    for (int round = 0; round < 2; ++round) {
        int ln = threadIdx.x & 63;
        asm volatile("" : "+v"(ln));
        char* const lw = lds + wave * 2048 + ln * 16;
        const char* const lr = lds + ln * 16;
        unsigned ti = li;
        if (round) { if (li >= (unsigned)(NH / 8)) break; ti = (unsigned)(NG / 8) + li; }
        const bool local = ti >= 24u;
        const unsigned pg = local ? (ti - 24u) >> 1 : ti / 6u;
        const unsigned pn = pg * 8u + (xg & 3u) * 2u + (ti & 1u);
        const unsigned rb = (local ? 3u : (ti % 6u) >> 1) * 2u + (xg >> 2);
        const unsigned rho = rb * 8u + (unsigned)wave;
        const unsigned rg = local ? rank * 16u + (rho - 48u) : remote_rg(rank, rho);
        unsigned ord = rank;
        {
            const bool fwd = pn >= 24u && ((pn >> 2) & 1u) != (rank & 1u);
            const unsigned tr = (pn - rank) & 7u;
#pragma unroll
            for (unsigned n = 1; n < 8; ++n) {
                unsigned s = (pn - (n - 1u < tr ? n - 1u : n)) & 7u;
                if (local) s = n == 7u ? rank ^ 1u : other_rank(rank, n - 1u);
                else if (fwd) s = n == 1u ? rank ^ 1u : other_rank(rank, n - 2u);
                ord |= s << (4u * n);
            }
        }

        f32x16 acc[4];
#pragma unroll
        for (int j = 0; j < 4; ++j)
#pragma unroll
            for (int e = 0; e < 16; ++e) acc[j][e] = 0.f;

        bool ok = true;
        const unsigned a_base = opb + OFF_LX + (rg << 14);
        const unsigned b_base = opb + OFF_LW + (pn << 16) + (unsigned)wave * 2048u;
        const unsigned lane16 = (unsigned)ln * 16u;
        const unsigned fl_off = (ln & 4) == 0 ? OFF_FX + rg * 128u + (unsigned)(ln & 3) * 8u + 4u
                                              : OFF_FW + pn * 128u + (unsigned)(((ln & 3) * 4 + (wave >> 1)) * 8) + 4u;
        const unsigned fl_str = (ln & 4) == 0 ? 128u * 128u : 32u * 128u;

        u32x4 A[4][4];
        u32x4 B[2][2];
        i32x8 bf[2][4];
#define LOAD_A(slot, rs, s, q) do { const unsigned _o = ((unsigned)(s) << 21) + (unsigned)(q) * 4096u + a_base; \
        _Pragma("unroll") for (int _u = 0; _u < 4; ++_u) A[slot][_u] = __builtin_amdgcn_raw_buffer_load_b128(rs, lane16, _o + _u * 1024u, AUX_SYS); } while (0)
#define LOAD_B(slot, rs, s, q) do { const unsigned _o = ((unsigned)(s) << 21) + (unsigned)(q) * 16384u + b_base; \
        _Pragma("unroll") for (int _u = 0; _u < 2; ++_u) B[slot][_u] = __builtin_amdgcn_raw_buffer_load_b128(rs, lane16, _o + _u * 1024u, AUX_SYS); } while (0)
#define WRITE_B(slot, buf) do { _Pragma("unroll") for (int _u = 0; _u < 2; ++_u) *(u32x4*)(lw + (buf) * 16384 + _u * 1024) = B[slot][_u]; } while (0)
#define READ_B(buf, t) do { \
            _Pragma("unroll") for (int _j = 0; _j < 4; ++_j) { \
                const u32x4 b0 = *(const u32x4*)(lr + (buf) * 16384 + (2 * (t)) * 4096 + _j * 1024); \
                const u32x4 b1 = *(const u32x4*)(lr + (buf) * 16384 + (2 * (t) + 1) * 4096 + _j * 1024); \
                _Pragma("unroll") for (int _e = 0; _e < 4; ++_e) { bf[t][_j][_e] = (int)b0[_e]; bf[t][_j][4 + _e] = (int)b1[_e]; } \
            } } while (0)
#define MMA(slot, t) do { \
            i32x8 af; \
            _Pragma("unroll") for (int _e = 0; _e < 4; ++_e) { af[_e] = (int)A[slot][2 * (t)][_e]; af[4 + _e] = (int)A[slot][2 * (t) + 1][_e]; } \
            _Pragma("unroll") for (int _j = 0; _j < 4; ++_j) \
                acc[_j] = __builtin_amdgcn_mfma_scale_f32_32x32x64_f8f6f4(af, bf[t][_j], acc[_j], 1, 1, 0, 0, 0, 0); \
        } while (0)
#define CHUNK(cs, rs, ls, lq, PRE) do { \
            READ_B((cs) & 1, 1); \
            SCHED(); \
            WRITE_B(((cs) + 1) & 1, ((cs) + 1) & 1); \
            SCHED(); \
            PRE; \
            LOAD_B(((cs) + 1) & 1, rs, ls, lq); LOAD_A(((cs) + 3) & 3, rs, ls, lq); \
            SCHED(); \
            MMA(cs, 0); \
            SCHED(); \
            __syncthreads(); \
            READ_B(((cs) + 1) & 1, 0); \
            SCHED(); \
            MMA(cs, 1); \
            SCHED(); \
        } while (0)

        unsigned s_cur = rank;
        ok &= wait_ge(ws, fl_off + s_cur * fl_str, call);
        LOAD_B(0, rl, s_cur, 0); LOAD_A(0, rl, s_cur, 0);
        LOAD_B(1, rl, s_cur, 1); LOAD_A(1, rl, s_cur, 1);
        __syncthreads();
        WRITE_B(0, 0);
        LOAD_B(0, rl, s_cur, 2); LOAD_A(2, rl, s_cur, 2);
        __syncthreads();
        READ_B(0, 0);
        for (int n = 0; n < 8; ++n) {
            const bool more = n < 7;
            ord >>= 4;
            const unsigned s_nxt = ord & 7u;
            const rsrc_t rn = mk_rsrc(ws, more ? OFF_PK : 0u);
            const unsigned fo = more ? fl_off + s_nxt * fl_str : OFF_TMO;
            unsigned fv;
            CHUNK(0, rl, s_cur, 3, fv = __builtin_amdgcn_raw_buffer_load_b32(rl, fo, 0, AUX_SYS); SCHED());
            CHUNK(1, rn, s_nxt, 0, asm volatile("" : "+v"(fv));
                               if (!more) ok &= !__any(fv == call);
                               else if (!__all(fv >= call)) ok &= wait_ge(ws, fo, call);
                               asm volatile("" ::: "memory"));
            CHUNK(2, rn, s_nxt, 1, );
            CHUNK(3, rn, s_nxt, 2, );
            s_cur = s_nxt;
        }

        const float scale = ok ? scale_ok : __builtin_nanf("");
        int tz = threadIdx.x & 63;
        asm volatile("" : "+v"(tz));
        if (local) {
            const rsrc_t ro = mk_rsrc(p.out, OUT_BYTES);
            const unsigned o_base = (((rg & 15u) * 32u + 4u * (unsigned)(tz >> 5)) * GN + (rank >> 1) * 4096u + pn * 128u + 4u * (unsigned)(tz & 31)) * 4u;
#pragma unroll
            for (int e = 0; e < 16; ++e) {
                u32x4 v;
#pragma unroll
                for (int j = 0; j < 4; ++j) {
                    const float y = acc[j][e] * scale;
                    v[j] = __builtin_bit_cast(unsigned, (y <= 0.f) ? 0.f : y);
                }
                const unsigned row = (unsigned)((e & 3) + 8 * (e >> 2));
                __builtin_amdgcn_raw_buffer_store_b128(v, ro, o_base + row * (GN * 4u), 0, AUX_SYS);
            }
        } else {
            const unsigned owner = rg >> 4, pid = (rg & 7u) * 32u + pn;
            unsigned tk = OFF_QP + owner * 128u;
            asm volatile("" : "+v"(tk));
            unsigned kq = 0;
            if (tz == 0) kq = __hip_atomic_fetch_add((unsigned*)(ws + tk), 1u, __ATOMIC_RELAXED, __HIP_MEMORY_SCOPE_AGENT);
            asm volatile("s_waitcnt lgkmcnt(0)" ::: "memory");
            __builtin_amdgcn_s_barrier();
            asm volatile("" ::: "memory");
            const unsigned sb0 = (unsigned)(wave * LSLOT) + 512u;
            const unsigned dummy = (unsigned)(wave * LSLOT) + (unsigned)tz * 4u;
            unsigned sb = sb0, mlo = 0, mhi = 0;
#pragma unroll
            for (int e = 0; e < 16; ++e) {
#pragma unroll
                for (int j = 0; j < 4; ++j) {
                    const float y = acc[j][e] * scale;
                    const bool nz = !(y <= 0.f);
                    const u64 m = __ballot(nz);
                    const unsigned pos = __builtin_amdgcn_mbcnt_hi((unsigned)(m >> 32), __builtin_amdgcn_mbcnt_lo((unsigned)m, 0u));
                    unsigned ad = (pos << 2) + sb;
                    asm("" : "+v"(ad));
                    *(__attribute__((address_space(3))) float*)(size_t)(nz ? ad : dummy) = y;
                    asm("s_nop 1\n\tv_writelane_b32 %0, %2, %4\n\tv_writelane_b32 %1, %3, %4"
                        : "+v"(mlo), "+v"(mhi) : "s"((unsigned)m), "s"((unsigned)(m >> 32)), "i"(e * 4 + j));
                    sb += (unsigned)__builtin_popcountll(m) << 2;
                }
            }
            *(u32x2*)(pk + tz * 8) = u32x2{mlo, mhi};
            const unsigned nbytes = (512u + (sb - sb0) + 63u) & ~63u;
            const rsrc_t rp = mk_rsrc(ws + OFF_PK + (owner * 256u + pid) * SLOT, nbytes);
            for (unsigned off = 0; off < nbytes; off += 1024u) {
                const u32x4 v = *(const u32x4*)(pk + off + tz * 16);
                __builtin_amdgcn_raw_buffer_store_b128(v, rp, (unsigned)tz * 16u + off, 0, AUX_SYS);
            }
            drain();
            asm volatile("" : "+v"(kq));
            if (tz == 0) store_flag(p.ws[owner] + OFF_Q + (rank * 256u + (kq & 255u)) * 64u, call, (pid << 16) | ((sb - sb0) >> 2));
        }
    }
}

extern "C" void kernel_launch(int rank, int nranks, ncclComm_t comm,
                              hipStream_t stream,
                              void* const* d_in, const int* in_sizes, int n_in,
                              void* d_out, int out_size,
                              const int* in_dtypes, int out_dtype,
                              void* const* d_out_all,
                              void* const* d_ws_all, size_t ws_bytes) {
    (void)comm; (void)in_sizes; (void)n_in; (void)d_out_all; (void)out_size; (void)in_dtypes; (void)out_dtype;
    if (nranks != NR || (unsigned)rank >= (unsigned)NR || ws_bytes < (size_t)WS_USED) {
        fprintf(stderr, "unsupported configuration\n");
        return;
    }
    Params p;
    p.x = (const unsigned char*)d_in[0];
    p.w = (const unsigned char*)d_in[1];
    p.sx = (const float*)d_in[2];
    p.sw = (const float*)d_in[3];
    p.out = (char*)d_out;
    for (int i = 0; i < NR; ++i) p.ws[i] = (char*)d_ws_all[i];
    p.rank = rank;
    HIP_CHECK(hipFuncSetAttribute((const void*)fused_gemm_rs, hipFuncAttributeMaxDynamicSharedMemorySize, (int)LDS_BYTES));
    hipLaunchKernelGGL(fused_gemm_rs, dim3(NWG), dim3(512), LDS_BYTES, stream, p);
    HIP_CHECK(hipGetLastError());
}
```

```cpp
#include <hip/hip_runtime.h>
#include <rccl/rccl.h>
#include <cstdio>
#include <cstdint>

#define HIP_CHECK(expr) do { hipError_t _e = (expr); if (_e != hipSuccess) { \
    fprintf(stderr, "HIP error %s at %s:%d\n", hipGetErrorString(_e), __FILE__, __LINE__); \
    return; } } while (0)

typedef unsigned u32x4 __attribute__((ext_vector_type(4)));
typedef unsigned u32x2 __attribute__((ext_vector_type(2)));
typedef int i32x8 __attribute__((ext_vector_type(8)));
typedef float f32x16 __attribute__((ext_vector_type(16)));
typedef unsigned long long u64;
typedef __amdgpu_buffer_rsrc_t rsrc_t;
typedef __attribute__((address_space(1))) unsigned gu32;
typedef __attribute__((address_space(1))) unsigned long long gu64;

constexpr int NR = 8;
constexpr int GN = 16384;
constexpr int KS = 512;
constexpr int NWG = 256;
constexpr int NH = 56;
constexpr int NG = NWG - NH;
constexpr unsigned NP0 = 48;
constexpr unsigned FU = 14;
constexpr u64 WDELAY = 150;
constexpr int POLL_SLEEP = 4;
constexpr unsigned NPIECE = 256;
constexpr unsigned X_BYTES = 4096u * KS;
constexpr unsigned W_BYTES = (unsigned)KS * GN;
constexpr unsigned OUT_BYTES = 512u * GN * 4u;
constexpr unsigned SLOT = 512u + 16384u;
constexpr unsigned LSLOT = 17408u;

constexpr unsigned OFF_LX = 0;
constexpr unsigned OFF_LW = 16u << 20;
constexpr unsigned OFF_SW = 32u << 20;
constexpr unsigned OPB = 40u << 20;
constexpr unsigned OFF_FX = 2u * OPB;
constexpr unsigned OFF_FW = OFF_FX + 8u * 128u * 128u;
constexpr unsigned JOFF = 8u * 128u * 128u + 8u * 32u * 128u;
constexpr unsigned OFF_Q = OFF_FX + 2u * JOFF;
constexpr unsigned OFF_CNT = OFF_Q + 8u * 256u * 64u;
constexpr unsigned OFF_TMO = OFF_CNT + 256u * 128u;
constexpr unsigned OFF_JTK = OFF_TMO + 128u;
constexpr unsigned OFF_QP = OFF_JTK + 8u * 128u;
constexpr unsigned OFF_PK = OFF_FX + (2u << 20);
constexpr unsigned WS_USED = OFF_PK + 8u * 256u * SLOT;
static_assert(OFF_QP + 8u * 128u <= OFF_PK, "layout");

constexpr u64 DEADLINE_TICKS = 100000000ull;
constexpr int AUX_SYS = 17;
constexpr unsigned LDS_BYTES = 8u * LSLOT;

struct Params {
    const unsigned char* x;
    const unsigned char* w;
    char* self;
    int rank, pad;
    const float* sx;
    const float* sw;
    char* out;
    char* mate;
    char* ws[NR];
    char* peer[NR];
};

__device__ __forceinline__ rsrc_t mk_rsrc(const void* p, unsigned bytes) {
    u64 a = (u64)p;
    unsigned lo = __builtin_amdgcn_readfirstlane((unsigned)a);
    unsigned hi = __builtin_amdgcn_readfirstlane((unsigned)(a >> 32));
    return __builtin_amdgcn_make_buffer_rsrc((void*)(((u64)hi << 32) | lo), 0, __builtin_amdgcn_readfirstlane(bytes), 0x00020000);
}

#define SCHED() __builtin_amdgcn_sched_barrier(0)

__device__ __forceinline__ bool wait_ge(char* ws, unsigned foff, unsigned call) {
    const gu32* f = (const gu32*)(ws + foff);
    gu32* tmo = (gu32*)(ws + OFF_TMO);
    unsigned v = __hip_atomic_load(f, __ATOMIC_RELAXED, __HIP_MEMORY_SCOPE_SYSTEM);
    if (__all(v >= call)) return true;
    const u64 t0 = __builtin_amdgcn_s_memrealtime();
    for (unsigned it = 0;; ++it) {
        __builtin_amdgcn_s_sleep(POLL_SLEEP);
        if (v < call) v = __hip_atomic_load(f, __ATOMIC_RELAXED, __HIP_MEMORY_SCOPE_SYSTEM);
        if (__all(v >= call)) return true;
        if ((it & 63u) != 63u) continue;
        const unsigned t = __builtin_amdgcn_readfirstlane(__hip_atomic_load(tmo, __ATOMIC_RELAXED, __HIP_MEMORY_SCOPE_AGENT));
        if (t == call || __builtin_amdgcn_s_memrealtime() - t0 > DEADLINE_TICKS) {
            __hip_atomic_store(tmo, call, __ATOMIC_RELAXED, __HIP_MEMORY_SCOPE_AGENT);
            return false;
        }
    }
}
__device__ __forceinline__ unsigned wait_flag64(char* ws, unsigned foff, unsigned call) {
    const gu64* f = (const gu64*)(ws + foff);
    gu32* tmo = (gu32*)(ws + OFF_TMO);
    const u64 t0 = __builtin_amdgcn_s_memrealtime();
    for (unsigned it = 0;; ++it) {
        const u64 v = __hip_atomic_load(f, __ATOMIC_RELAXED, __HIP_MEMORY_SCOPE_SYSTEM);
        const unsigned hi = __builtin_amdgcn_readfirstlane((unsigned)(v >> 32));
        const unsigned lo = __builtin_amdgcn_readfirstlane((unsigned)v);
        if (hi >= call) return lo;
        __builtin_amdgcn_s_sleep(POLL_SLEEP);
        if ((it & 63u) != 63u) continue;
        const unsigned t = __builtin_amdgcn_readfirstlane(__hip_atomic_load(tmo, __ATOMIC_RELAXED, __HIP_MEMORY_SCOPE_AGENT));
        if (t == call || __builtin_amdgcn_s_memrealtime() - t0 > DEADLINE_TICKS) {
            __hip_atomic_store(tmo, call, __ATOMIC_RELAXED, __HIP_MEMORY_SCOPE_AGENT);
            return ~0u;
        }
    }
}

__device__ __forceinline__ void drain() {
    asm volatile("" ::: "memory");
    __builtin_amdgcn_s_waitcnt(0x0F70);
    asm volatile("s_waitcnt vmcnt(0)" ::: "memory");
}
__device__ __forceinline__ void store_flag(char* addr, unsigned call, unsigned payload) {
    __hip_atomic_store((gu64*)addr, ((u64)call << 32) | payload, __ATOMIC_RELAXED, __HIP_MEMORY_SCOPE_SYSTEM);
}

__device__ __forceinline__ unsigned other_rank(unsigned c, unsigned i) { return ((((c >> 1) + 1u + (i >> 1)) & 3u) << 1) | (i & 1u); }
__device__ __forceinline__ unsigned remote_rg(unsigned c, unsigned rho) { return other_rank(c, rho % 6u) * 16u + (c & 1u) * 8u + rho / 6u; }

__device__ __forceinline__ void load_w_item(const unsigned char* w, unsigned col0, unsigned u, int lane, unsigned (&v)[16]) {
    const rsrc_t rw = mk_rsrc(w, W_BYTES);
    const unsigned src = (32u * u + 16u * (unsigned)(lane >> 5)) * GN + col0 + 4u * (unsigned)(lane & 31);
#pragma unroll
    for (int i = 0; i < 16; ++i) v[i] = __builtin_amdgcn_raw_buffer_load_b32(rw, src + (unsigned)i * GN, 0, 0);
}
__device__ __forceinline__ void order_w_item(const unsigned (&v)[16], u32x4 (&d)[4]) {
#pragma unroll
    for (int q = 0; q < 4; ++q) {
        const unsigned lo01 = __builtin_amdgcn_perm(v[4 * q + 1], v[4 * q], 0x05010400u);
        const unsigned hi01 = __builtin_amdgcn_perm(v[4 * q + 1], v[4 * q], 0x07030602u);
        const unsigned lo23 = __builtin_amdgcn_perm(v[4 * q + 3], v[4 * q + 2], 0x05010400u);
        const unsigned hi23 = __builtin_amdgcn_perm(v[4 * q + 3], v[4 * q + 2], 0x07030602u);
        d[0][q] = __builtin_amdgcn_perm(lo23, lo01, 0x05040100u);
        d[1][q] = __builtin_amdgcn_perm(lo23, lo01, 0x07060302u);
        d[2][q] = __builtin_amdgcn_perm(hi23, hi01, 0x05040100u);
        d[3][q] = __builtin_amdgcn_perm(hi23, hi01, 0x07060302u);
    }
}
__device__ __forceinline__ void load_x_item(const unsigned char* x, unsigned nb, unsigned rg, unsigned q, int lane, u32x4 (&d)[4]) {
    const rsrc_t rx = mk_rsrc(x, nb);
    const unsigned src = (32u * rg + (unsigned)(lane & 31)) * KS + 16u * (unsigned)(lane >> 5) + 128u * q;
#pragma unroll
    for (int r = 0; r < 4; ++r) d[r] = __builtin_amdgcn_raw_buffer_load_b128(rx, src + 32u * r, 0, 0);
}

__global__ void __launch_bounds__(512, 2) fused_gemm_rs(Params p) {
    extern __shared__ __attribute__((aligned(16))) char lds[];
    const int wave = __builtin_amdgcn_readfirstlane(threadIdx.x >> 6);
    unsigned rank = (unsigned)p.rank;
    const int b = blockIdx.x;
    char* ws = p.self;
    const unsigned char* px = p.x;
    const unsigned char* pw = p.w;
    char* wsp = p.peer[b % 7];
    asm volatile("" : "+s"(ws), "+s"(px), "+s"(pw), "+s"(rank), "+s"(wsp));

    gu32* cnt = (gu32*)(ws + OFF_CNT + (unsigned)b * 128u + (unsigned)wave * 4u);

    if (b < NH) {
        int lane = threadIdx.x & 63;
        asm volatile("" : "+v"(lane));
        const int c32 = lane & 31, h = lane >> 5;
        unsigned cnt_v = __hip_atomic_load(cnt, __ATOMIC_RELAXED, __HIP_MEMORY_SCOPE_AGENT);
        const unsigned peer = (rank + 1u + (unsigned)(b % 7)) & 7u;
        const bool partner = peer == (rank ^ 1u);
        char* const wsq = p.mate;
        const unsigned j = (unsigned)(b / 7);
        const unsigned l16 = (unsigned)lane * 16u;
        const bool push = (wave & 3) != 3;
        u32x4 dp[4];
        const unsigned pm = j * 6u + (unsigned)((wave >> 2) * 3 + (wave & 3));
        const unsigned prg = remote_rg(peer, pm >> 2), pq = pm & 3u;
        load_x_item(px, push ? X_BYTES : 0u, prg, pq, lane, dp);
        asm volatile("" : "+v"(cnt_v) :: "memory");
        const unsigned call = __builtin_amdgcn_readfirstlane(cnt_v) + 1u;
        unsigned pfl = push ? OFF_FX + (rank * 128u + prg) * 128u + pq * 8u : 0u;
        {
            const rsrc_t rd = mk_rsrc(wsp + (call & 1u) * OPB + OFF_LX + (rank << 21) + (prg << 14) + (pq << 12), push ? 4096u : 0u);
#pragma unroll
            for (int r = 0; r < 4; ++r) __builtin_amdgcn_raw_buffer_store_b128(dp[r], rd, (unsigned)r * 1024u + l16, 0, AUX_SYS);
        }
        if (lane == 0) __hip_atomic_store(cnt, call, __ATOMIC_RELAXED, __HIP_MEMORY_SCOPE_AGENT);

        constexpr unsigned PBUF = 24576u;
        const unsigned opb = (call & 1u) * OPB;
        constexpr unsigned XO_P = 704u + 24u * FU, XO_N = 640u + 4u * (16u - FU);
        const unsigned nij = partner ? (XO_P + 64u - NP0) / 2u : (XO_N + 64u - NP0) / 2u;
        const unsigned njob = partner ? nij : nij + NPIECE;
        const rsrc_t rl = mk_rsrc(ws, OFF_PK);
        unsigned tka = OFF_JTK + peer * 128u;
        asm volatile("" : "+v"(tka));
        const unsigned tbase = (call - 1u) * njob - 32u;
        unsigned k0 = j, k1 = j + 8u, k2 = j + 16u;
        unsigned tkv = j + 24u + tbase;
        u32x4 d[3];
        unsigned so1 = 0, do0 = 0, do1 = 0, fo0 = 0, fo1 = 0, so2 = 0, do2 = 0, fo2 = 0, jnb;
#define JOB_DESC(k, SO, DO, FO) do { \
            if ((k) < nij) { \
                const unsigned _m = NP0 + 2u * (k) + (unsigned)(wave >> 2), _sub = (unsigned)(wave & 3) * 1024u; \
                const unsigned _xo = partner ? XO_P : XO_N; \
                if (_m < 192u || _m >= _xo) { \
                    const unsigned _rg = _m < 192u ? remote_rg(rank, _m >> 2) : rank * 16u + ((_m - _xo) >> 2), _q = _m & 3u; \
                    SO = DO = opb + OFF_LX + (peer << 21) + (_rg << 14) + (_q << 12) + _sub; \
                    FO = OFF_FX + (peer * 128u + _rg) * 128u + _q * 8u; \
                } else { \
                    unsigned _pn, _s = peer, _fw = 0u, _u = _m & 15u; \
                    if (_m < 576u) _pn = ((_m - 192u) >> 7) * 8u + (((_m >> 4) + 4u + peer) & 7u); \
                    else if (!partner) { \
                        if (_m < 640u) { _pn = 24u + (rank & 1u) * 4u + (((_m >> 4) + peer) & 3u); _fw = _u < FU ? 0x80000000u : 0u; } \
                        else { _pn = 28u - (rank & 1u) * 4u + (_m & 3u); _u = FU + ((_m - 640u) >> 2); } \
                    } \
                    else if (_m < 704u) _pn = 24u + (((rank & 1u) * 4u + ((_m - 576u) >> 4)) & 7u); \
                    else { \
                        const unsigned _f = (_m - 704u) / FU; \
                        _u = (_m - 704u) % FU; _pn = 28u - (rank & 1u) * 4u + (_f & 3u); _s = other_rank(rank, _f >> 2); \
                    } \
                    const unsigned _o = (_pn << 16) + (_u << 12); \
                    DO = opb + OFF_LW + (_s << 21) + _o + _sub; \
                    SO = partner ? DO : opb + OFF_SW + ((rank >> 1) << 21) + _o + _sub; \
                    FO = (OFF_FW + _s * 4096u + (_o >> 9)) | _fw; \
                } \
            } else { \
                FO = OFF_Q - JOFF + (peer * 256u + (((k) - nij) & 255u)) * 64u; \
            } \
            SO = __builtin_amdgcn_readfirstlane(SO); DO = __builtin_amdgcn_readfirstlane(DO); FO = __builtin_amdgcn_readfirstlane(FO); \
        } while (0)
#define READY_FLAG(FO) (((FO) & 0x7fffffffu) + JOFF)
#define PIECE_BYTES(e) (((e) & 0xffffu) <= 4096u ? (512u + ((e) & 0xffffu) * 4u + 63u) & ~63u : 0u)
#define JOB_SRC(k, e, SO) do { \
            jnb = 1024u; \
            if ((k) >= nij) { \
                const unsigned _pb = PIECE_BYTES(e); \
                SO = OFF_PK + (rank * 256u + (((e) >> 16) & 255u)) * SLOT + (unsigned)wave * 1024u; \
                jnb = _pb > (unsigned)wave * 1024u ? _pb - (unsigned)wave * 1024u : 0u; \
            } \
            SO = __builtin_amdgcn_readfirstlane(SO); jnb = __builtin_amdgcn_readfirstlane(jnb); \
            asm volatile("" : "+s"(SO), "+s"(jnb));        \
        } while (0)
#define JOB_LOADS(SO) do { \
            const rsrc_t _rs = mk_rsrc(wsp + SO, jnb); \
            _Pragma("unroll") for (int _r = 0; _r < 3; ++_r) d[_r] = __builtin_amdgcn_raw_buffer_load_b128(_rs, l16 + _r * 8192u, 0, AUX_SYS); \
        } while (0)
#define JOB_NEXT(BLOCK_BELOW) do { \
            JOB_DESC(k2, so2, do2, fo2); \
            asm volatile("" : "+s"(fo2)); \
            enx = ~0u; \
            if (k1 < njob) { \
                enx = __builtin_amdgcn_readfirstlane(fv[0]); \
                if (__builtin_amdgcn_readfirstlane(fv[1]) < call) \
                    enx = k1 < (BLOCK_BELOW) ? wait_flag64(ws, READY_FLAG(fo1), call) : ~0u; \
            } \
            JOB_SRC(k1, enx, so1); \
        } while (0)
#define JOB_ADVANCE() do { \
            if (threadIdx.x == 0) { \
                *(unsigned*)(lds + 2 * PBUF + buf * 4u) = tkv;             \
                tkv = __hip_atomic_fetch_add((gu32*)(ws + tka), 1u, __ATOMIC_RELAXED, __HIP_MEMORY_SCOPE_AGENT); \
            } \
            fv = __builtin_amdgcn_raw_buffer_load_b64(rl, READY_FLAG(fo2), 0, AUX_SYS); \
            SCHED(); \
            JOB_LOADS(so1); \
            asm volatile("s_waitcnt lgkmcnt(0)" ::: "memory"); \
            __builtin_amdgcn_s_barrier(); \
            asm volatile("" ::: "memory"); \
        } while (0)
#define JOB_ROTATE() do { \
            const unsigned _tkn = *(const unsigned*)(lds + 2 * PBUF + buf * 4u); \
            ent = enx; k0 = k1; k1 = k2; do0 = do1; fo0 = fo1; so1 = so2; do1 = do2; fo1 = fo2; \
            k2 = __builtin_amdgcn_readfirstlane(_tkn) - tbase; \
            buf ^= 1u; \
        } while (0)
#define PUBLISH(FO) do { \
            if ((FO) && (threadIdx.x & 255) < 2u) { \
                if ((threadIdx.x & 1u) == 0u) store_flag(ws + ((FO) & 0x7fffffffu), call, 0u); \
                else if ((FO) & 0x80000000u) store_flag(wsq + READY_FLAG(FO), call, 0u); \
            } \
        } while (0)
        unsigned so0 = 0;
        JOB_DESC(k0, so0, do0, fo0);
        JOB_DESC(k1, so1, do1, fo1);
        if ((wave & 3) == 3) wait_flag64(ws, READY_FLAG(fo0), call);
        asm volatile("" ::: "memory");
        __builtin_amdgcn_s_barrier();
        asm volatile("" ::: "memory");
        unsigned ent = 0, enx;
        u32x2 fv = __builtin_amdgcn_raw_buffer_load_b64(rl, READY_FLAG(fo1), 0, AUX_SYS);
        JOB_SRC(k0, ent, so0);
        SCHED();
        JOB_LOADS(so0);
        unsigned buf = 0;
        unsigned pend = 0;
        while (k0 < nij) {
            JOB_NEXT(nij);
            drain();
            asm volatile("" :: "v"(d[1]), "v"(d[2]));
            if (pfl) {
                if (lane == 0) store_flag(wsp + pfl, call, 0u);
                pfl = 0;
            }
            __builtin_amdgcn_raw_buffer_store_b128(d[0], rl, do0 + l16, 0, AUX_SYS);
            JOB_ADVANCE();
            PUBLISH(pend);
            pend = fo0;
            JOB_ROTATE();
        }
        drain();
        __builtin_amdgcn_s_barrier();
        asm volatile("" ::: "memory");
        PUBLISH(pend);
        if (k0 < njob && ent == ~0u) {
            ent = wait_flag64(ws, READY_FLAG(fo0), call);
            JOB_SRC(k0, ent, so0);
            JOB_LOADS(so0);
            drain();
        }
        const unsigned o_peer = ((peer & 1u) * 256u + 4u * (unsigned)h) * GN + (peer >> 1) * 4096u + 4u * (unsigned)c32;
        unsigned lastp = 0, bad = 0;
        while (k0 < njob) {
            char* const pb = lds + buf * PBUF;
            const unsigned nb = PIECE_BYTES(ent);
            JOB_NEXT(njob);
            SCHED();
#pragma unroll
            for (int r = 0; r < 3; ++r) *(u32x4*)(pb + wave * 1024 + l16 + r * 8192u) = d[r];
            JOB_ADVANCE();
            const u32x2 mk = *(const u32x2*)(pb + lane * 8);
            const rsrc_t ro = mk_rsrc(p.out, nb ? OUT_BYTES : 0u);
            const unsigned pid = (ent >> 16) & 255u;
            if (nb) lastp = pid; else bad = 1u;
            unsigned pc = lane < wave * 8 ? (unsigned)__builtin_popcount(mk[0]) + (unsigned)__builtin_popcount(mk[1]) : 0u;
            pc += (unsigned)__builtin_amdgcn_update_dpp(0, (int)pc, 0x111, 0xf, 0xf, false);
            pc += (unsigned)__builtin_amdgcn_update_dpp(0, (int)pc, 0x112, 0xf, 0xf, false);
            pc += (unsigned)__builtin_amdgcn_update_dpp(0, (int)pc, 0x114, 0xf, 0xf, false);
            pc += (unsigned)__builtin_amdgcn_update_dpp(0, (int)pc, 0x118, 0xf, 0xf, false);
            const unsigned pre = __builtin_amdgcn_readlane(pc, 15) + __builtin_amdgcn_readlane(pc, 31) +
                                 __builtin_amdgcn_readlane(pc, 47) + __builtin_amdgcn_readlane(pc, 63);
            unsigned sb = buf * PBUF + 512u + (pre << 2);
            const unsigned o_base = (o_peer + (pid >> 5) * 32u * GN + (pid & 31u) * 128u) * 4u;
#pragma unroll
            for (int ee = 0; ee < 2; ++ee) {
                u32x4 o;
#pragma unroll
                for (int jj = 0; jj < 4; ++jj) {
                    const unsigned lo = __builtin_amdgcn_readlane(mk[0], wave * 8 + ee * 4 + jj);
                    const unsigned hi = __builtin_amdgcn_readlane(mk[1], wave * 8 + ee * 4 + jj);
                    const u64 m = ((u64)hi << 32) | lo;
                    const unsigned pos = __builtin_amdgcn_mbcnt_hi(hi, __builtin_amdgcn_mbcnt_lo(lo, 0u));
                    const unsigned val = *(const unsigned*)(lds + ((pos << 2) + sb));
                    o[jj] = __builtin_amdgcn_inverse_ballot_w64(m) ? val : 0u;
                    sb += (unsigned)__builtin_popcountll(m) << 2;
                }
                const unsigned e = (unsigned)(wave * 2 + ee);
                const unsigned row = (e & 3u) + 8u * (e >> 2);
                __builtin_amdgcn_raw_buffer_store_b128(o, ro, o_base + row * (GN * 4u), 0, AUX_SYS);
            }
            JOB_ROTATE();
        }
        if (bad) {
            drain();
            const unsigned e = (unsigned)wave * 2u;
            if (lane == 0) ((float*)p.out)[o_peer + ((lastp >> 5) * 32u + (e & 3u) + 8u * (e >> 2)) * GN + (lastp & 31u) * 128u] = __builtin_nanf("");
        }
        return;
    }

    const int g = b - NH;
    unsigned call = 0;
    float scale_ok;
    {
        const int lane = threadIdx.x & 63;
        unsigned cnt_v = __hip_atomic_load(cnt, __ATOMIC_RELAXED, __HIP_MEMORY_SCOPE_AGENT);
        const rsrc_t rd = mk_rsrc(ws, OFF_FX);
        const unsigned c = (unsigned)lane & 7u;
        if (g >= 64) {
            const u64 t0 = __builtin_amdgcn_s_memrealtime();
            while (__builtin_amdgcn_s_memrealtime() - t0 < WDELAY) __builtin_amdgcn_s_sleep(2);
        }
        for (unsigned jx = (unsigned)(g * 8 + wave); jx < 2560u; jx += NG * 8) {
            u32x4 d[4];
            unsigned dst, fo;
            u64 fa;
            bool need;
#define STAGE_MID() do { \
                unsigned cz = c; \
                asm volatile("" : "+v"(cz), "+v"(cnt_v) :: "memory"); \
                fa = (u64)p.ws[0]; \
                _Pragma("unroll") for (unsigned i = 1; i < 8; ++i) fa = cz == i ? (u64)p.ws[i] : fa; \
                fa += fo + (cz == rank ? 0u : JOFF); \
                call = __builtin_amdgcn_readfirstlane(cnt_v) + 1u; \
                asm volatile("" ::: "memory"); \
            } while (0)
#define STAGE_END() do { \
                dst += (call & 1u) * OPB + (unsigned)lane * 16u; \
                _Pragma("unroll") for (int r = 0; r < 4; ++r) __builtin_amdgcn_raw_buffer_store_b128(d[r], rd, dst + (unsigned)r * 1024u, 0, AUX_SYS); \
                asm volatile("; stage end %0" :: "i"(__LINE__) : "memory"); \
            } while (0)
            if (jx < 512u) {
                const unsigned rg = (jx >> 2 & 7u) * 16u + (jx >> 5 & 1u) * 8u + (jx >> 6), q = jx & 3u;
                load_x_item(px, X_BYTES, rg, q, lane, d);
                dst = OFF_LX + (rank << 21) + (rg << 14) + (q << 12);
                fo = OFF_FX + (rank * 128u + rg) * 128u + q * 8u;
                need = c == (rg >> 4) || ((c >> 1) != (rg >> 5) && (c & 1u) == ((rg >> 3) & 1u));
                STAGE_MID();
                STAGE_END();
            } else {
                const unsigned wi = jx - 512u, grp = wi & 3u, u = (wi >> 2) & 15u, ps = wi >> 6;
                const unsigned pn = (ps & 24u) + ((ps + rank) & 7u);
                unsigned v[16];
                load_w_item(pw, grp * 4096u + pn * 128u, u, lane, v);
                const unsigned o = (pn << 16) + (u << 12);
                const bool own = grp == (rank >> 1);
                dst = (own ? OFF_LW + (rank << 21) : OFF_SW + (grp << 21)) + o;
                fo = OFF_FW + rank * 4096u + (o >> 9);
                need = (c >> 1) == grp && (own || pn < 24u || u >= FU || ((pn >> 2) & 1u) == (c & 1u));
                STAGE_MID();
                order_w_item(v, d);
                STAGE_END();
            }
            drain();
            if (lane < 8 && need) store_flag((char*)fa, call, 0u);
        }
        if (lane == 0) __hip_atomic_store(cnt, call, __ATOMIC_RELAXED, __HIP_MEMORY_SCOPE_AGENT);
        scale_ok = __builtin_bit_cast(float, __builtin_amdgcn_readfirstlane(__builtin_bit_cast(unsigned, p.sx[0] * p.sw[0])));
    }
    const rsrc_t rl = mk_rsrc(ws, OFF_PK);
    const unsigned xg = (unsigned)b & 7u, li = (unsigned)g >> 3;
    const unsigned opb = (call & 1u) * OPB;
    char* const pk = lds + wave * LSLOT;

    for (int round = 0; round < 2; ++round) {
        int ln = threadIdx.x & 63;
        asm volatile("" : "+v"(ln));
        char* const lw = lds + wave * 2048 + ln * 16;
        const char* const lr = lds + ln * 16;
        unsigned ti = li;
        if (round) { if (li >= (unsigned)(NH / 8)) break; ti = (unsigned)(NG / 8) + li; }
        const bool local = ti >= 24u;
        const unsigned pg = local ? (ti - 24u) >> 1 : ti / 6u;
        const unsigned pn = pg * 8u + (xg & 3u) * 2u + (ti & 1u);
        const unsigned rb = (local ? 3u : (ti % 6u) >> 1) * 2u + (xg >> 2);
        const unsigned rho = rb * 8u + (unsigned)wave;
        const unsigned rg = local ? rank * 16u + (rho - 48u) : remote_rg(rank, rho);
        unsigned ord = rank;
        {
            const bool fwd = pn >= 24u && ((pn >> 2) & 1u) != (rank & 1u);
            const unsigned tr = (pn - rank) & 7u;
#pragma unroll
            for (unsigned n = 1; n < 8; ++n) {
                unsigned s = (pn - (n - 1u < tr ? n - 1u : n)) & 7u;
                if (local) s = n == 7u ? rank ^ 1u : other_rank(rank, n - 1u);
                else if (fwd) s = n == 1u ? rank ^ 1u : other_rank(rank, n - 2u);
                ord |= s << (4u * n);
            }
        }

        f32x16 acc[4];
#pragma unroll
        for (int j = 0; j < 4; ++j)
#pragma unroll
            for (int e = 0; e < 16; ++e) acc[j][e] = 0.f;

        bool ok = true;
        const unsigned a_base = opb + OFF_LX + (rg << 14);
        const unsigned b_base = opb + OFF_LW + (pn << 16) + (unsigned)wave * 2048u;
        const unsigned lane16 = (unsigned)ln * 16u;
        const unsigned fl_off = (ln & 4) == 0 ? OFF_FX + rg * 128u + (unsigned)(ln & 3) * 8u + 4u
                                              : OFF_FW + pn * 128u + (unsigned)(((ln & 3) * 4 + (wave >> 1)) * 8) + 4u;
        const unsigned fl_str = (ln & 4) == 0 ? 128u * 128u : 32u * 128u;

        u32x4 A[4][4];
        u32x4 B[2][2];
        i32x8 bf[2][4];
#define LOAD_A(slot, rs, s, q) do { const unsigned _o = ((unsigned)(s) << 21) + (unsigned)(q) * 4096u + a_base; \
        _Pragma("unroll") for (int _u = 0; _u < 4; ++_u) A[slot][_u] = __builtin_amdgcn_raw_buffer_load_b128(rs, lane16, _o + _u * 1024u, AUX_SYS); } while (0)
#define LOAD_B(slot, rs, s, q) do { const unsigned _o = ((unsigned)(s) << 21) + (unsigned)(q) * 16384u + b_base; \
        _Pragma("unroll") for (int _u = 0; _u < 2; ++_u) B[slot][_u] = __builtin_amdgcn_raw_buffer_load_b128(rs, lane16, _o + _u * 1024u, AUX_SYS); } while (0)
#define WRITE_B(slot, buf) do { _Pragma("unroll") for (int _u = 0; _u < 2; ++_u) *(u32x4*)(lw + (buf) * 16384 + _u * 1024) = B[slot][_u]; } while (0)
#define READ_B(buf, t) do { \
            _Pragma("unroll") for (int _j = 0; _j < 4; ++_j) { \
                const u32x4 b0 = *(const u32x4*)(lr + (buf) * 16384 + (2 * (t)) * 4096 + _j * 1024); \
                const u32x4 b1 = *(const u32x4*)(lr + (buf) * 16384 + (2 * (t) + 1) * 4096 + _j * 1024); \
                _Pragma("unroll") for (int _e = 0; _e < 4; ++_e) { bf[t][_j][_e] = (int)b0[_e]; bf[t][_j][4 + _e] = (int)b1[_e]; } \
            } } while (0)
#define MMA(slot, t) do { \
            i32x8 af; \
            _Pragma("unroll") for (int _e = 0; _e < 4; ++_e) { af[_e] = (int)A[slot][2 * (t)][_e]; af[4 + _e] = (int)A[slot][2 * (t) + 1][_e]; } \
            _Pragma("unroll") for (int _j = 0; _j < 4; ++_j) \
                acc[_j] = __builtin_amdgcn_mfma_scale_f32_32x32x64_f8f6f4(af, bf[t][_j], acc[_j], 1, 1, 0, 0, 0, 0); \
        } while (0)
#define CHUNK(cs, rs, ls, lq, PRE) do { \
            READ_B((cs) & 1, 1); \
            SCHED(); \
            WRITE_B(((cs) + 1) & 1, ((cs) + 1) & 1); \
            SCHED(); \
            PRE; \
            LOAD_B(((cs) + 1) & 1, rs, ls, lq); LOAD_A(((cs) + 3) & 3, rs, ls, lq); \
            SCHED(); \
            MMA(cs, 0); \
            SCHED(); \
            __syncthreads(); \
            READ_B(((cs) + 1) & 1, 0); \
            SCHED(); \
            MMA(cs, 1); \
            SCHED(); \
        } while (0)

        unsigned s_cur = rank;
        ok &= wait_ge(ws, fl_off + s_cur * fl_str, call);
        asm volatile("" ::: "memory");
        LOAD_B(0, rl, s_cur, 0); LOAD_A(0, rl, s_cur, 0);
        LOAD_B(1, rl, s_cur, 1); LOAD_A(1, rl, s_cur, 1);
        __syncthreads();
        WRITE_B(0, 0);
        LOAD_B(0, rl, s_cur, 2); LOAD_A(2, rl, s_cur, 2);
        __syncthreads();
        READ_B(0, 0);
        for (int n = 0; n < 8; ++n) {
            const bool more = n < 7;
            ord >>= 4;
            const unsigned s_nxt = ord & 7u;
            const rsrc_t rn = mk_rsrc(ws, more ? OFF_PK : 0u);
            const unsigned fo = more ? fl_off + s_nxt * fl_str : OFF_TMO;
            unsigned fv;
            CHUNK(0, rl, s_cur, 3, fv = __builtin_amdgcn_raw_buffer_load_b32(rl, fo, 0, AUX_SYS); SCHED());
            CHUNK(1, rn, s_nxt, 0, asm volatile("" : "+v"(fv));
                               if (!more) ok &= !__any(fv == call);
                               else if (!__all(fv >= call)) ok &= wait_ge(ws, fo, call);
                               asm volatile("" ::: "memory"));
            CHUNK(2, rn, s_nxt, 1, );
            CHUNK(3, rn, s_nxt, 2, );
            s_cur = s_nxt;
        }

        const float scale = ok ? scale_ok : __builtin_nanf("");
        int tz = threadIdx.x & 63;
        asm volatile("" : "+v"(tz));
        if (local) {
            const rsrc_t ro = mk_rsrc(p.out, OUT_BYTES);
            const unsigned o_base = (((rg & 15u) * 32u + 4u * (unsigned)(tz >> 5)) * GN + (rank >> 1) * 4096u + pn * 128u + 4u * (unsigned)(tz & 31)) * 4u;
#pragma unroll
            for (int e = 0; e < 16; ++e) {
                u32x4 v;
#pragma unroll
                for (int j = 0; j < 4; ++j) {
                    const float y = acc[j][e] * scale;
                    v[j] = __builtin_bit_cast(unsigned, (y <= 0.f) ? 0.f : y);
                }
                const unsigned row = (unsigned)((e & 3) + 8 * (e >> 2));
                __builtin_amdgcn_raw_buffer_store_b128(v, ro, o_base + row * (GN * 4u), 0, AUX_SYS);
            }
        } else {
            const unsigned owner = rg >> 4, pid = (rg & 7u) * 32u + pn;
            unsigned tk = OFF_QP + owner * 128u;
            asm volatile("" : "+v"(tk));
            unsigned kq = 0;
            if (tz == 0) kq = __hip_atomic_fetch_add((gu32*)(ws + tk), 1u, __ATOMIC_RELAXED, __HIP_MEMORY_SCOPE_AGENT);
            asm volatile("s_waitcnt lgkmcnt(0)" ::: "memory");
            __builtin_amdgcn_s_barrier();
            asm volatile("" ::: "memory");
            const unsigned sb0 = (unsigned)(wave * LSLOT) + 512u;
            const unsigned dummy = (unsigned)(wave * LSLOT) + (unsigned)tz * 4u;
            unsigned sb = sb0, mlo = 0, mhi = 0;
#pragma unroll
            for (int e = 0; e < 16; ++e) {
#pragma unroll
                for (int j = 0; j < 4; ++j) {
                    const float y = acc[j][e] * scale;
                    const bool nz = !(y <= 0.f);
                    const u64 m = __ballot(nz);
                    const unsigned pos = __builtin_amdgcn_mbcnt_hi((unsigned)(m >> 32), __builtin_amdgcn_mbcnt_lo((unsigned)m, 0u));
                    unsigned ad = (pos << 2) + sb;
                    asm("" : "+v"(ad));
                    *(__attribute__((address_space(3))) float*)(size_t)(nz ? ad : dummy) = y;
                    asm("s_nop 1\n\tv_writelane_b32 %0, %2, %4\n\tv_writelane_b32 %1, %3, %4"
                        : "+v"(mlo), "+v"(mhi) : "s"((unsigned)m), "s"((unsigned)(m >> 32)), "i"(e * 4 + j));
                    sb += (unsigned)__builtin_popcountll(m) << 2;
                }
            }
            asm volatile("" ::: "memory");
            *(u32x2*)(pk + tz * 8) = u32x2{mlo, mhi};
            asm volatile("" ::: "memory");
            const unsigned nbytes = (512u + (sb - sb0) + 63u) & ~63u;
            const rsrc_t rp = mk_rsrc(ws + OFF_PK + (owner * 256u + pid) * SLOT, nbytes);
            for (unsigned off = 0; off < nbytes; off += 1024u) {
                const u32x4 v = *(const u32x4*)(pk + off + tz * 16);
                __builtin_amdgcn_raw_buffer_store_b128(v, rp, (unsigned)tz * 16u + off, 0, AUX_SYS);
            }
            drain();
            asm volatile("" : "+v"(kq));
            if (tz == 0) store_flag(p.ws[owner] + OFF_Q + (rank * 256u + (kq & 255u)) * 64u, call, (pid << 16) | ((sb - sb0) >> 2));
        }
    }
}

extern "C" void kernel_launch(int rank, int nranks, ncclComm_t comm,
                              hipStream_t stream,
                              void* const* d_in, const int* in_sizes, int n_in,
                              void* d_out, int out_size,
                              const int* in_dtypes, int out_dtype,
                              void* const* d_out_all,
                              void* const* d_ws_all, size_t ws_bytes) {
    (void)comm; (void)in_sizes; (void)n_in; (void)d_out_all; (void)out_size; (void)in_dtypes; (void)out_dtype;
    if (nranks != NR || (unsigned)rank >= (unsigned)NR || ws_bytes < (size_t)WS_USED) {
        fprintf(stderr, "unsupported configuration\n");
        return;
    }
    Params p;
    p.x = (const unsigned char*)d_in[0];
    p.w = (const unsigned char*)d_in[1];
    p.sx = (const float*)d_in[2];
    p.sw = (const float*)d_in[3];
    p.out = (char*)d_out;
    for (int i = 0; i < NR; ++i) p.ws[i] = (char*)d_ws_all[i];
    for (int i = 0; i < NR; ++i) p.peer[i] = p.ws[(rank + 1 + i) & 7];
    p.self = p.ws[rank];
    p.mate = p.ws[rank ^ 1];
    p.rank = rank;
    p.pad = 0;
    HIP_CHECK(hipFuncSetAttribute((const void*)fused_gemm_rs, hipFuncAttributeMaxDynamicSharedMemorySize, (int)LDS_BYTES));
    hipLaunchKernelGGL(fused_gemm_rs, dim3(NWG), dim3(512), LDS_BYTES, stream, p);
    HIP_CHECK(hipGetLastError());
}
```
